```python
import numpy as np

kernel_name = "gpu_ag_gemm_m4096_k4096_n4096_f32_relu"


ABSMAX_RTOL = 0.001

COMPUTE_DTYPE = "float32"
EPILOGUE = "relu"
IN_QUANT = None
OUT_QUANT = None
IN_DTYPES = {"x": "float32", "w_mat": "float32"}
WORLD_SIZE = 8

def setup_inputs():

    x = np.random.randn(4096, 4096).astype(np.float32)
    w_mat = (np.random.randn(4096, 4096) / np.sqrt(4096)).astype(np.float32)
    return {"x": x, "w_mat": w_mat}

def _epilogue(y):
    return np.maximum(y, 0.0).astype(np.float32)

def reference(x, w_mat):
    y = (x @ w_mat).astype(np.float32)
    return _epilogue(y)
```

```cpp
#include <hip/hip_runtime.h>
#include <rccl/rccl.h>
#include <algorithm>
#include <cstdio>
#include <type_traits>
#include <vector>

#define HIP_CHECK(expr) do { hipError_t _e = (expr); if (_e != hipSuccess) { \
    fprintf(stderr, "HIP error %s at %s:%d\n", hipGetErrorString(_e), __FILE__, __LINE__); \
    return; } } while (0)

namespace {

#define CFG_KU 24
#define CFG_DEPTH 3
#define CFG_DEPTH_IN CFG_DEPTH
#define CFG_CLAIM_AT 8
#define CFG_TOKEN_AT 9
#define CFG_YGATE 24
#define CFG_YFLUSH 2
#define CFG_DELAY 48
#define CFG_STAGGER 12
#define CFG_NLOOP 24
#define CFG_NPRE 4
#define CFG_PSLEEP 2
#define CFG_WSLEEP 8
#define CFG_POLLAUX 17
#define CFG_LDAUX 17

#define PNL_OFF 0
#define CFG_LPACE 4
#define CFG_YSPAN 1700
#define CFG_YBURST 2
#define NOB_OFF 0

constexpr int NRANK = 8;
constexpr int KDIM = 4096, MSH = 512, NSH = 512;
constexpr int KU = CFG_KU, GU = 32 - KU;
#define CFG_GOWN 5
#define CFG_GH {6, 6, 7, 9}
constexpr int GOWN = CFG_GOWN, NQUAD = 48, NHQ = NQUAD - 4 * GOWN;
constexpr int NOWN_U = 32 + 3 * KU + 4 * GU, NHLP_U = 4 * KU + 4 * GU;
constexpr int UNIT_B = 16 * KDIM * 2, ITEM_B = 16384;
constexpr int NIT_IN = 8 * KU, NIT_X = 64 + 8 * GU;

constexpr int XA_OFF = 0, XA_PAR = NRANK * 32 * UNIT_B;
constexpr int WB_OFF = XA_OFF + 2 * XA_PAR, WB_PAR = 4 * 4 * 16 * ITEM_B;
constexpr int XF_OFF = WB_OFF + 2 * WB_PAR, XF_PAR = NRANK * 32 * 128;
constexpr int WF_OFF = XF_OFF + 2 * XF_PAR, WF_PAR = 16 * 128;
constexpr int YF_OFF = WF_OFF + 2 * WF_PAR;
constexpr int CNT_OFF = YF_OFF + 4 * 512;
constexpr int TMO_OFF = CNT_OFF + 1024 * 4;
constexpr int QC_OFF = TMO_OFF + 1024;
constexpr int DBG_OFF = TMO_OFF + 4096;
constexpr int WS_USED = DBG_OFF + 1024 * 64 * 8;

constexpr int X_BYTES = MSH * KDIM * 4, W_BYTES = KDIM * NSH * 4, OUT_BYTES = NRANK * MSH * NSH * 4;
static_assert(X_BYTES == W_BYTES, "one descriptor size serves both inputs");
constexpr int RSRC_W3 = 0x00020000;
constexpr int AUX_SYS = 17;
constexpr int OOB = 0x7ffffff0;
constexpr unsigned long long DEADLINE_TICKS = 50000000ull;

typedef unsigned v4u __attribute__((ext_vector_type(4)));
typedef unsigned v2u __attribute__((ext_vector_type(2)));
typedef float f32x4 __attribute__((ext_vector_type(4)));
typedef float f32x2 __attribute__((ext_vector_type(2)));
typedef _Float16 h8 __attribute__((ext_vector_type(8)));
typedef _Float16 h2 __attribute__((ext_vector_type(2)));
typedef __amdgpu_buffer_rsrc_t rsrc_t;

struct Args {
  const float* x;
  const float* w;
  char* ws[NRANK];
  float* out[NRANK];
  int rank;
  unsigned quad[NQUAD];
  unsigned char list[2][160];
};

#define LAS __attribute__((address_space(3)))
#define SCHED_FENCE() __builtin_amdgcn_sched_barrier(0)
#define STR2(x) #x
#define STR(x) STR2(x)
#define WAIT_VM(n) asm volatile("s_waitcnt vmcnt(" STR(n) ")" ::: "memory")
#define WAIT_LGKM0() asm volatile("s_waitcnt lgkmcnt(0)" ::: "memory")

__device__ __forceinline__ rsrc_t mk_rsrc(const void* p, int bytes) {
  return __builtin_amdgcn_make_buffer_rsrc((void*)p, 0, bytes, RSRC_W3);
}
__device__ __forceinline__ unsigned rfl(unsigned v) { return (unsigned)__builtin_amdgcn_readfirstlane((int)v); }
__device__ __forceinline__ unsigned pk_f16(float a, float b) {
  h2 v = {(_Float16)a, (_Float16)b};
  return __builtin_bit_cast(unsigned, v);
}
__device__ __forceinline__ f32x4 mfma(v4u a, v4u b, f32x4 c) {
  return __builtin_amdgcn_mfma_f32_16x16x32_f16(__builtin_bit_cast(h8, a), __builtin_bit_cast(h8, b), c, 0, 0, 0);
}
template <int CTRL, int ROWMASK>
__device__ __forceinline__ unsigned dpp_max(unsigned m) {
  const unsigned o = (unsigned)__builtin_amdgcn_update_dpp((int)m, (int)m, CTRL, ROWMASK, 0xf, false);
  return o > m ? o : m;
}

__device__ __forceinline__ int block_scale(const v4u (&f)[32]) {
  float m = 0.f;
#pragma unroll
  for (int i = 0; i < 32; ++i) {
    const f32x4 v = __builtin_bit_cast(f32x4, f[i]);
    m = fmaxf(m, fmaxf(fmaxf(fabsf(v.x), fabsf(v.y)), fmaxf(fabsf(v.z), fabsf(v.w))));
  }
  unsigned u = __builtin_bit_cast(unsigned, m);
  u = dpp_max<0xB1, 0xf>(u);
  u = dpp_max<0x4E, 0xf>(u);
  u = dpp_max<0x141, 0xf>(u);
  u = dpp_max<0x128, 0xf>(u);
  const unsigned m0 = (unsigned)__builtin_amdgcn_readlane((int)u, 0), m1 = (unsigned)__builtin_amdgcn_readlane((int)u, 16);
  const unsigned m2 = (unsigned)__builtin_amdgcn_readlane((int)u, 32), m3 = (unsigned)__builtin_amdgcn_readlane((int)u, 48);
  const unsigned mm = max(max(m0, m1), max(m2, m3));
  int se = 127 + 14 + 127 - (int)(mm >> 23);
  return se > 254 ? 254 : se;
}

template <int PACE = 0>
__device__ __forceinline__ void load_block(v4u (&f)[32], rsrc_t rs, int vo, int vd, int so, int s0, int s1) {
#pragma unroll
  for (int m = 0; m < 16; ++m) {
    f[2 * m] = __builtin_amdgcn_raw_buffer_load_b128(rs, vo, so + (m >> 2) * s1 + (m & 3) * s0, 0);
    f[2 * m + 1] = __builtin_amdgcn_raw_buffer_load_b128(rs, vo + vd, so + (m >> 2) * s1 + (m & 3) * s0, 0);
    if (PACE > 0) {
      SCHED_FENCE();
      __builtin_amdgcn_s_sleep(PACE);
      SCHED_FENCE();
    }
  }
  SCHED_FENCE();
}
constexpr int X_VD = 16, X_S0 = KDIM * 4, X_S1 = 4 * KDIM * 4;
__device__ __forceinline__ int x_vo(int lane) { return lane * 32; }
__device__ __forceinline__ int x_so(int unit, int seg) { return unit * (16 * KDIM * 4) + seg * 2048; }
constexpr int W_VD = NSH * 4, W_S0 = 2 * NSH * 4, W_S1 = 64 * NSH * 4;
__device__ __forceinline__ int w_vo(int lane) { return (((lane >> 3) & 1) * 32 + (lane >> 4) * 8) * (NSH * 4) + (lane & 7) * 16; }
__device__ __forceinline__ int w_so(int chunk, int col0) { return chunk * (256 * NSH * 4) + col0 * 4; }

__device__ __forceinline__ int cvt_x(const v4u (&f)[32], v4u (&h)[16], char* lds, int lane) {
  const int se = block_scale(f);
  const float sc = __builtin_bit_cast(float, (unsigned)se << 23);
#pragma unroll
  for (int r = 0; r < 16; ++r) {
    const f32x4 a = __builtin_bit_cast(f32x4, f[2 * r]) * sc, b = __builtin_bit_cast(f32x4, f[2 * r + 1]) * sc;
    const v4u p = {pk_f16(a.x, a.y), pk_f16(a.z, a.w), pk_f16(b.x, b.y), pk_f16(b.z, b.w)};
    *(v4u*)(lds + lane * 256 + ((r ^ (lane & 15)) << 4)) = p;
  }
  const int q = lane >> 4, r = lane & 15;
#pragma unroll
  for (int s = 0; s < 16; ++s) {
    const int l = 4 * s + q;
    h[s] = *(const v4u*)(lds + l * 256 + ((r ^ (l & 15)) << 4));
  }
  return se;
}

__device__ __forceinline__ int cvt_w(const v4u (&f)[32], v4u (&h)[16]) {
  const int se = block_scale(f);
  const float sc = __builtin_bit_cast(float, (unsigned)se << 23);
#pragma unroll
  for (int t = 0; t < 4; ++t) {
    v4u p[4];
#pragma unroll
    for (int jj = 0; jj < 4; ++jj) {
      const f32x4 a = __builtin_bit_cast(f32x4, f[8 * t + 2 * jj]) * sc, b = __builtin_bit_cast(f32x4, f[8 * t + 2 * jj + 1]) * sc;
      p[0][jj] = pk_f16(a.x, b.x);
      p[1][jj] = pk_f16(a.y, b.y);
      p[2][jj] = pk_f16(a.z, b.z);
      p[3][jj] = pk_f16(a.w, b.w);
    }
#pragma unroll
    for (int c = 0; c < 2; ++c)
#pragma unroll
      for (int jj = 0; jj < 4; ++jj) {
        h[4 * t + c][jj] = (unsigned)__builtin_amdgcn_update_dpp((int)p[c][jj], (int)p[c + 2][jj], 0x128, 0xf, 0xc, false);
        h[4 * t + 2 + c][jj] = (unsigned)__builtin_amdgcn_update_dpp((int)p[c + 2][jj], (int)p[c][jj], 0x128, 0xf, 0x3, false);
      }
  }
  return se;
}

struct Waiter {
  rsrc_t rs;
  unsigned seq;
  unsigned long long t0;
  unsigned spins;
  bool dead;
  __device__ __forceinline__ bool give_up() {
    if (dead) return true;
    if ((++spins & 63u) != 0u) return false;
    const unsigned t = rfl(__builtin_amdgcn_raw_buffer_load_b32(rs, 0, TMO_OFF, AUX_SYS));
    if (t == seq || __builtin_amdgcn_s_memrealtime() - t0 > DEADLINE_TICKS) {
      __builtin_amdgcn_raw_buffer_store_b32(seq, rs, 0, TMO_OFF, AUX_SYS);
      dead = true;
      return true;
    }
    return false;
  }
};

#define STAMP(i) do { } while (0)
#define STAMPV(i, v) do { } while (0)
#define STAMP_OUT() do { } while (0)
#define STAMP_DECL() do { } while (0)

constexpr int RED_WAVE = 16 * 144, RED_BUF = 4 * RED_WAVE;
constexpr int YMAX = 30;
constexpr int LDS_TOKEN = 2 * RED_BUF + 4 * YMAX * 512;
constexpr int LDS_BYTES = LDS_TOKEN + 64;
static_assert(LDS_TOKEN >= 4 * 16384, "the transpose areas lie below the token");

__global__ void __launch_bounds__(256, 1) ag_gemm_kernel(const Args a) {
  __shared__ __attribute__((aligned(16))) char lds[LDS_BYTES];
  const int lane = threadIdx.x & 63, w = (int)rfl(threadIdx.x >> 6);
  const int rank = a.rank, myidx = rank & 3, gbase = rank & 4, obase = gbase ^ 4;
  const int xcd = blockIdx.x & 7, slot = blockIdx.x >> 3;
  const rsrc_t rsS = mk_rsrc(a.ws[rank], WS_USED);
  const rsrc_t rsX = mk_rsrc(a.x, X_BYTES), rsW = mk_rsrc(a.w, W_BYTES);
  const int vone = lane == 0 ? 0 : OOB;
  char* const wlds = lds + w * 16384;
  LAS int* const token = (LAS int*)((LAS char*)lds + LDS_TOKEN);
  STAMP_DECL();
  STAMP(0);

  const int cnt_off = CNT_OFF + (blockIdx.x * 4 + w) * 4;
  const unsigned cnt_v = __builtin_amdgcn_raw_buffer_load_b32(rsS, 0, cnt_off, AUX_SYS);
  Waiter wt;
  wt.rs = rsS;
  wt.t0 = __builtin_amdgcn_s_memrealtime();
  wt.spins = 0;
  wt.dead = false;
#define TAKE_SEQ() \
  const unsigned seq = rfl(cnt_v) + 1u; \
  __builtin_amdgcn_raw_buffer_store_b32(seq, rsS, vone, cnt_off, AUX_SYS); \
  const int par = (int)(seq & 1u); \
  wt.seq = seq

  if (slot < 7) {
    const bool cross = slot >= 3;
    const unsigned peer = (unsigned)(cross ? obase + ((myidx + slot - 3) & 3) : gbase + ((myidx + 1 + slot) & 3));
    const int nit = cross ? NIT_X : NIT_IN;
    const rsrc_t rsD = mk_rsrc(a.ws[peer & 7u], WS_USED);
    v4u f[32], h[16];
    auto issue = [&](int n, auto pace) {
      const bool lw = cross && n < 64;
      const rsrc_t rs = mk_rsrc(lw ? a.w : a.x, X_BYTES);
      const int vo = n < nit ? (lw ? w_vo(lane) : x_vo(lane)) : 0x08000000;
      const int so = lw ? w_so(n & 15, (int)(peer & 3u) * 128 + (n >> 4) * 32) : x_so(((cross ? KU - 8 : 0) + (n >> 3)) & 31, n & 7);
      load_block<decltype(pace)::value>(f, rs, PNL_OFF + vo, lw ? W_VD : X_VD, so, lw ? W_S0 : X_S0, lw ? W_S1 : X_S1);
    };
    int vzero = 0;
    asm volatile("" : "+v"(vzero));
    int n = w * 8 + xcd;
    if (threadIdx.x == 0) *token = 0;
    WAIT_LGKM0();
    __builtin_amdgcn_s_barrier();
    for (int i = 0; i < w; ++i) __builtin_amdgcn_s_sleep(CFG_STAGGER);
    issue(n, std::integral_constant<int, 0>());
    TAKE_SEQ();
    const unsigned qbase = (seq - 1u) * (unsigned)nit - 32u;
    for (int turn = w; n < nit; turn += 4) {
      const bool isw = cross && n < 64;
      const int unit = (cross ? KU - 8 : 0) + (n >> 3);
      int se;
      if (isw) se = cvt_w(f, h);
      else se = cvt_x(f, h, wlds, lane);
      const int doff = isw ? WB_OFF + par * WB_PAR + (myidx * 64 + n) * ITEM_B
                           : XA_OFF + par * XA_PAR + (rank * 32 + unit) * UNIT_B + (n & 7) * ITEM_B;
      const int foff = isw ? WF_OFF + par * WF_PAR + (myidx * 4 + (n >> 4)) * 128 + (n & 15) * 4
                           : XF_OFF + par * XF_PAR + (rank * 32 + unit) * 128 + (n & 7) * 4;
      WAIT_LGKM0();
      if (turn < 4) STAMP(40);
      while ((int)rfl(__hip_atomic_load(token, __ATOMIC_RELAXED, __HIP_MEMORY_SCOPE_WORKGROUP)) < turn) __builtin_amdgcn_s_sleep(1);
      SCHED_FENCE();
      int claim = 0;
#pragma unroll
      for (int s = 0; s < 16; ++s) {
        if (s == CFG_CLAIM_AT && lane == 0) claim = __builtin_amdgcn_raw_ptr_buffer_atomic_add_i32(1, rsS, vzero, QC_OFF + slot * 128, 0);
        __builtin_amdgcn_raw_buffer_store_b128(h[s], rsD, lane * 16 + (s & 3) * 1024, doff + (s >> 2) * 4096, AUX_SYS);
        if (s == CFG_TOKEN_AT) __hip_atomic_fetch_max(token, turn + 1, __ATOMIC_RELAXED, __HIP_MEMORY_SCOPE_WORKGROUP);
        if (s < 15) WAIT_VM(CFG_DEPTH);
        SCHED_FENCE();
      }
      STAMPV(2 + 2 * ((turn >> 2) & 15), n);
      const unsigned nn = rfl((unsigned)claim) - qbase;
      n = nn < (unsigned)nit ? (int)nn : nit;
      WAIT_VM(0);
      __builtin_amdgcn_raw_buffer_store_b32(seq * 256u + (unsigned)se, rsD, vone, foff, AUX_SYS);
      SCHED_FENCE();
      STAMP(3 + 2 * ((turn >> 2) & 15));
      if (n >= nit) break;
      issue(n, std::integral_constant<int, CFG_LPACE>());
      __builtin_amdgcn_raw_buffer_store_b32(0u, rsD, OOB, 0, AUX_SYS);
    }
    __hip_atomic_fetch_max(token, 0x7fffffff, __ATOMIC_RELAXED, __HIP_MEMORY_SCOPE_WORKGROUP);
    STAMP_OUT();
    return;
  }

  if (slot == 7) {
    TAKE_SEQ();
    (void)par;
    if (xcd != 0) return;
    const int vo = lane < NHQ ? lane * 16 : OOB;
    for (;;) {
      const v4u v = __builtin_amdgcn_raw_buffer_load_b128(rsS, vo, YF_OFF + w * 512, AUX_SYS);
      const bool ok = lane >= NHQ || (v.x >= seq && v.y >= seq && v.z >= seq && v.w >= seq);
      if (__all(ok)) break;
      if (wt.give_up()) break;
      __builtin_amdgcn_s_sleep(2);
    }
    if (wt.dead) {
      const rsrc_t rsO = mk_rsrc(a.out[rank], OUT_BYTES);
      __builtin_amdgcn_raw_buffer_store_b32(0x7fc00000u, rsO, vone, rank * (MSH * NSH * 4), AUX_SYS);
    }
    STAMP(1);
    STAMP_OUT();
    return;
  }

  const int cj = slot - 8, hsrc = cj & 3;
  const unsigned qd = a.quad[xcd * 6 + (cj >> 2)];
  const bool helper = (qd & 4u) != 0u;
  const int hpan = (int)(qd & 3u), grp = (int)((qd >> 8) & 255u), ngrp = (int)((qd >> 16) & 255u), hq = (int)(qd >> 24);
  unsigned lst;
  int nun;
  {
    const int li = grp + lane * ngrp;
    const bool inr = li < (helper ? NHLP_U : NOWN_U);
    const unsigned e = a.list[helper ? 1 : 0][inr ? li : 0];
    const bool keep = inr && !(helper && (int)(e >> 5) == obase + hsrc);
    const unsigned long long km = __ballot(keep);
    nun = __builtin_popcountll(km);
    const int pos = keep ? (int)__builtin_amdgcn_mbcnt_hi((unsigned)(km >> 32), __builtin_amdgcn_mbcnt_lo((unsigned)km, 0u)) : 63;
    lst = (unsigned)__builtin_amdgcn_ds_permute(pos * 4, (int)e);
  }
  const unsigned dest = (unsigned)(helper ? obase + hsrc : rank);
  const int col0 = helper ? myidx * 128 + hpan * 32 : (hpan * 4 + hsrc) * 32;
  const rsrc_t rsO = mk_rsrc(a.out[dest & 7u], OUT_BYTES);

  v4u b[64];
  v4u av[32];
  int eb[4];
  TAKE_SEQ();
  __builtin_amdgcn_s_sleep(CFG_DELAY);
  if (helper) {
    const int hid = (hq * 4 + hsrc) * 4 + w;
    if (hid < 256) {
      v4u h[16];
      const int unit = hid >> 3, seg = hid & 7;
      load_block(av, rsX, x_vo(lane), X_VD, x_so(unit, seg), X_S0, X_S1);
      const int se = cvt_x(av, h, wlds, lane);
      const int doff = XA_OFF + par * XA_PAR + (rank * 32 + unit) * UNIT_B + seg * ITEM_B;
#pragma unroll
      for (int s = 0; s < 16; ++s)
        __builtin_amdgcn_raw_buffer_store_b128(h[s], rsS, lane * 16 + (s & 3) * 1024, doff + (s >> 2) * 4096, AUX_SYS);
      WAIT_VM(0);
      __builtin_amdgcn_raw_buffer_store_b32(seq * 256u + (unsigned)se, rsS, vone,
                                            XF_OFF + par * XF_PAR + (rank * 32 + unit) * 128 + seg * 4, AUX_SYS);
    }
    STAMP(1);
    const int fo = WF_OFF + par * WF_PAR + (hsrc * 4 + hpan) * 128 + w * 16;
    unsigned f0, f1, f2, f3;
    for (;;) {
      const v4u v = __builtin_amdgcn_raw_buffer_load_b128(rsS, 0, fo, CFG_POLLAUX);
      f0 = rfl(v.x); f1 = rfl(v.y); f2 = rfl(v.z); f3 = rfl(v.w);
      if ((f0 >> 8) == seq && (f1 >> 8) == seq && (f2 >> 8) == seq && (f3 >> 8) == seq) break;
      if (wt.give_up()) break;
      __builtin_amdgcn_s_sleep(CFG_WSLEEP);
    }
    eb[0] = f0 & 255; eb[1] = f1 & 255; eb[2] = f2 & 255; eb[3] = f3 & 255;
    STAMP(2);
    const int bo = WB_OFF + par * WB_PAR + ((hsrc * 4 + hpan) * 16 + w * 4) * ITEM_B;
#pragma unroll
    for (int s = 0; s < 64; ++s)
      b[s] = __builtin_amdgcn_raw_buffer_load_b128(rsS, NOB_OFF + lane * 16 + (s & 3) * 1024, bo + (s >> 2) * 4096, AUX_SYS);
    SCHED_FENCE();
  } else {
    v4u fb[32], h[16];
    auto ldw = [&](v4u (&f)[32], int j) { load_block(f, rsW, NOB_OFF + w_vo(lane), W_VD, w_so(w * 4 + j, col0), W_S0, W_S1); };
    auto cvw = [&](const v4u (&f)[32], auto jt) {
      constexpr int j = decltype(jt)::value;
      eb[j] = cvt_w(f, h);
#pragma unroll
      for (int s = 0; s < 16; ++s) b[16 * j + s] = h[s];
      SCHED_FENCE();
    };
    ldw(av, 0);
    ldw(fb, 1);
    cvw(av, std::integral_constant<int, 0>());
    ldw(av, 2);
    cvw(fb, std::integral_constant<int, 1>());
    ldw(fb, 3);
    cvw(av, std::integral_constant<int, 2>());
    cvw(fb, std::integral_constant<int, 3>());
  }
  __syncthreads();
  STAMP(3);

  const int xa = XA_OFF + par * XA_PAR + w * (2 * ITEM_B);
  const int xf = XF_OFF + par * XF_PAR + w * 8;
  const int va = lane * 16;
  auto entry = [&](int i) { return __builtin_amdgcn_readlane((int)lst, i < nun ? i : nun - 1); };
  auto poll = [&](int e) { return __builtin_amdgcn_raw_buffer_load_b64(rsS, 0, xf + e * 128, CFG_POLLAUX); };
  auto load_a = [&](int s, int e) {
    return __builtin_amdgcn_raw_buffer_load_b128(rsS, va + (s & 3) * 1024, xa + e * UNIT_B + (s >> 2) * 4096, CFG_LDAUX);
  };
  const int vy = (w * 4 + ((lane >> 3) & 3)) * (NSH * 4) + (lane & 7) * 16;
  char* const yb = lds + 2 * RED_BUF + w * (YMAX * 512);
  const int qoff = QC_OFF + (3 + ((hsrc - myidx) & 3)) * 128;
  const unsigned qgate = seq * (unsigned)NIT_X - 32u + CFG_YGATE;
  int ydone = 0, yhave = 0;
  bool ygo = !helper;
  auto gate_open = [&](unsigned qv) { return (int)(qv - qgate) >= 0; };
  unsigned ynext = 0;
  const int ypace = 2 * CFG_YSPAN / nun;
  auto now32 = [&]() { return (unsigned)__builtin_amdgcn_s_memrealtime(); };
  auto open_gate = [&]() { ygo = true; ynext = now32() + (unsigned)(((hq * 4 + w) * 37 & 127) * ypace >> 7); };
  auto may_flush = [&]() {
    if (!ygo || ydone >= yhave) return false;
    if (!helper) return true;
    const unsigned t = now32();
    if ((int)(t - ynext) < 0) return false;
    ynext = (int)(t - ynext) > CFG_YBURST * ypace ? t - (CFG_YBURST - 1) * ypace : ynext + ypace;
    return true;
  };
  auto flush2 = [&]() {
    const int e0 = entry(ydone), e1 = entry(ydone + 1);
    const v4u yv = *(const v4u*)(yb + ydone * 512 + lane * 16);
    const int vo = lane < 32 ? vy + e0 * (16 * NSH * 4) : (ydone + 1 < yhave ? vy + e1 * (16 * NSH * 4) : OOB);
    __builtin_amdgcn_raw_buffer_store_b128(yv, rsO, vo, col0 * 4, AUX_SYS);
    ydone = ydone + 2 < yhave ? ydone + 2 : yhave;
  };
  unsigned g0 = 0, g1 = 0;
  auto wait_unit = [&](int e) {
    for (;;) {
      const v2u v = poll(e);
      if (!ygo && gate_open(rfl(__builtin_amdgcn_raw_buffer_load_b32(rsS, 0, qoff, CFG_POLLAUX)))) open_gate();
      g0 = rfl(v.x); g1 = rfl(v.y);
      if ((g0 >> 8) == seq && (g1 >> 8) == seq) break;
      if (wt.give_up()) break;
      if (may_flush()) flush2();
      else __builtin_amdgcn_s_sleep(CFG_PSLEEP);
    }
  };

  const int red_w = (lane & 15) * 144 + ((lane >> 4) & 1) * 64 + (lane >> 5) * 8;
  const int red_r = (w * 4 + ((lane >> 3) & 3)) * 144 + (lane & 7) * 16;

  const int ecur = entry(0);
  wait_unit(ecur);
  v2u pn = poll(entry(1));
  unsigned pqn = __builtin_amdgcn_raw_buffer_load_b32(rsS, 0, qoff, CFG_POLLAUX);
#pragma unroll
  for (int s = 0; s < 32; ++s) av[s] = load_a(s, ecur);
  SCHED_FENCE();

  auto unit_body = [&](auto pf_tag, int i, int enext, int ea0, int ea1) __attribute__((always_inline)) {
    constexpr bool PF = decltype(pf_tag)::value;
    constexpr int NLOOP = CFG_NLOOP, NPRE = CFG_NPRE;
    const int ex[4] = {254 - ea0 - eb[0], 254 - ea0 - eb[1], 254 - ea1 - eb[2], 254 - ea1 - eb[3]};
    const float bad = wt.dead ? __builtin_nanf("") : 0.f;
    f32x4 acc[4][2];
    f32x2 p[4];
#pragma unroll
    for (int j = 0; j < 4; ++j) {
      acc[j][0] = acc[j][1] = f32x4{0.f, 0.f, 0.f, 0.f};
      p[j] = f32x2{bad, bad};
    }
#pragma unroll
    for (int s = 0; s < 32; ++s) {
      const v4u cur = av[s];
      acc[s >> 3][0] = mfma(b[2 * s], cur, acc[s >> 3][0]);
      acc[s >> 3][1] = mfma(b[2 * s + 1], cur, acc[s >> 3][1]);
      if (PF && (s + 1) * NLOOP / 32 > s * NLOOP / 32) av[s * NLOOP / 32] = load_a(s * NLOOP / 32, enext);
      if (s >= 8 && (s & 7) >= 2 && (s & 7) < 6) {
        const int ch = (s >> 3) - 1, j = (s & 7) - 2;
        p[j][0] += __builtin_amdgcn_ldexpf(acc[ch][0][j], ex[ch]);
        p[j][1] += __builtin_amdgcn_ldexpf(acc[ch][1][j], ex[ch]);
        asm volatile("" : "+v"(p[j]));
      }
      SCHED_FENCE();
    }
    if (!PF) pn = poll(enext);
    char* const rb = lds + (i & 1) * RED_BUF;
#pragma unroll
    for (int j = 0; j < 4; ++j) {
      if (PF && j < NPRE) av[NLOOP + j] = load_a(NLOOP + j, enext);
      SCHED_FENCE();
      p[j][0] += __builtin_amdgcn_ldexpf(acc[3][0][j], ex[3]);
      p[j][1] += __builtin_amdgcn_ldexpf(acc[3][1][j], ex[3]);
      *(f32x2*)(rb + w * RED_WAVE + red_w + j * 16) = p[j];
      SCHED_FENCE();
    }
    WAIT_LGKM0();
    __builtin_amdgcn_s_barrier();
    SCHED_FENCE();
    const f32x4 p0 = *(const f32x4*)(rb + red_r), p1 = *(const f32x4*)(rb + RED_WAVE + red_r);
    const f32x4 p2 = *(const f32x4*)(rb + 2 * RED_WAVE + red_r), p3 = *(const f32x4*)(rb + 3 * RED_WAVE + red_r);
    SCHED_FENCE();
    if (PF) {
#pragma unroll
      for (int s = NLOOP + NPRE; s < 32; ++s) av[s] = load_a(s, enext);
    }
    SCHED_FENCE();
    f32x4 y = (p0 + p1) + (p2 + p3);
    y.x = y.x <= 0.f ? 0.f : y.x;
    y.y = y.y <= 0.f ? 0.f : y.y;
    y.z = y.z <= 0.f ? 0.f : y.z;
    y.w = y.w <= 0.f ? 0.f : y.w;
    *(f32x4*)(yb + i * 512 + (lane & 31) * 16) = y;
    SCHED_FENCE();
  };

  for (int i = 0; i < nun; ++i) {
    const int ea0 = (int)(g0 & 255u), ea1 = (int)(g1 & 255u);
    const int enext = entry(i + 1);
    const unsigned n0 = rfl(pn.x), n1 = rfl(pn.y), pq = rfl(pqn);
    const bool more = i + 1 < nun;
    const bool pf = more && (n0 >> 8) == seq && (n1 >> 8) == seq;
    if (pf) {
      pn = poll(entry(i + 2));
      pqn = __builtin_amdgcn_raw_buffer_load_b32(rsS, 0, qoff, CFG_POLLAUX);
      SCHED_FENCE();
      unit_body(std::true_type(), i, enext, ea0, ea1);
      g0 = n0; g1 = n1;
    } else {
      unit_body(std::false_type(), i, enext, ea0, ea1);
      g0 = rfl(pn.x); g1 = rfl(pn.y);
    }
    yhave = i + 1;
    if (!ygo && gate_open(pq)) open_gate();
    while ((!pf || yhave - ydone >= 2) && may_flush()) flush2();
    SCHED_FENCE();
    STAMP(4 + i);
    if (more && !pf) {
      if ((g0 >> 8) != seq || (g1 >> 8) != seq) wait_unit(enext);
      pn = poll(entry(i + 2));
      pqn = __builtin_amdgcn_raw_buffer_load_b32(rsS, 0, qoff, CFG_POLLAUX);
#pragma unroll
      for (int s = 0; s < 32; ++s) av[s] = load_a(s, enext);
      SCHED_FENCE();
    }
  }
  while (!ygo) {
    if (gate_open(rfl(__builtin_amdgcn_raw_buffer_load_b32(rsS, 0, qoff, CFG_POLLAUX))) || wt.give_up()) open_gate();
    __builtin_amdgcn_s_sleep(CFG_PSLEEP);
  }
  STAMPV(42, yhave - ydone);
  while (ydone < yhave) {
    if (may_flush()) flush2();
    else __builtin_amdgcn_s_sleep(1);
  }
  if (helper) {
    WAIT_VM(0);
    const rsrc_t rsD = mk_rsrc(a.ws[dest & 7u], WS_USED);
    __builtin_amdgcn_raw_buffer_store_b32(seq, rsD, vone, YF_OFF + myidx * 512 + (hq * 4 + w) * 4, AUX_SYS);
  }
  STAMP(40);
  STAMPV(41, (int)qd);
  STAMP_OUT();
}

struct Ev { double t; int e; };

}

extern "C" void kernel_launch(int rank, int nranks, ncclComm_t comm,
                              hipStream_t stream,
                              void* const* d_in, const int* in_sizes, int n_in,
                              void* d_out, int out_size,
                              const int* in_dtypes, int out_dtype,
                              void* const* d_out_all,
                              void* const* d_ws_all, size_t ws_bytes) {
  (void)comm; (void)in_dtypes; (void)out_dtype; (void)d_out;
  if (nranks != NRANK || n_in < 2 || (unsigned)rank >= (unsigned)NRANK || ws_bytes < (size_t)WS_USED ||
      in_sizes[0] != MSH * KDIM || in_sizes[1] != KDIM * NSH || out_size != NRANK * MSH * NSH) {
    fprintf(stderr, "unexpected problem shape\n");
    return;
  }
  Args a = {};
  a.x = (const float*)d_in[0];
  a.w = (const float*)d_in[1];
  for (int p = 0; p < NRANK; ++p) {
    a.ws[p] = (char*)d_ws_all[p];
    a.out[p] = (float*)d_out_all[p];
  }
  a.rank = rank;

  const int gbase = rank & 4, obase = gbase ^ 4;
  std::vector<Ev> ev;
  for (int u = 0; u < 32; ++u) ev.push_back({0.001 * u, rank * 32 + u});
  for (int p = gbase; p < gbase + 4; ++p)
    if (p != rank)
      for (int u = 0; u < KU; ++u) ev.push_back({1.0 + u + 0.01 * p, p * 32 + u});
  for (int p = obase; p < obase + 4; ++p)
    for (int v = 0; v < GU; ++v) ev.push_back({9.1 + v + 0.01 * p, p * 32 + KU + v});
  std::stable_sort(ev.begin(), ev.end(), [](const Ev& l, const Ev& r) { return l.t < r.t; });
  int n0 = 0, n1 = 0;
  for (const Ev& v : ev) {
    a.list[0][n0++] = (unsigned char)v.e;
    if (!((v.e >> 5) == rank && (v.e & 31) >= KU)) a.list[1][n1++] = (unsigned char)v.e;
  }
  const int gh[4] = CFG_GH;
  static_assert(NHQ <= 32 && NHQ >= 16, "done flags of one helper fit 512 bytes; the helpers convert 256 blocks");
  if (gh[0] + gh[1] + gh[2] + gh[3] != NHQ || n0 != NOWN_U || n1 != NHLP_U) { fprintf(stderr, "bad quad split\n"); return; }
  int used[8] = {0, 0, 0, 0, 0, 0, 0, 0};
  for (int c = 0; c < 4; ++c)
    for (int g = 0; g < GOWN; ++g) {
      const int x = c + 4 * (g & 1);
      a.quad[x * 6 + used[x]++] = (unsigned)c | (unsigned)g << 8 | (unsigned)GOWN << 16;
    }
  int hq = 0, x = 0;
  for (int p = 3; p >= 0; --p)
    for (int g = 0; g < gh[p]; ++g) {
      if ((NHLP_U + gh[p] - 1) / gh[p] > YMAX) { fprintf(stderr, "bad quad split\n"); return; }
      while (used[x] >= 6) x = (x + 1) & 7;
      a.quad[x * 6 + used[x]++] = (unsigned)(4 + p) | (unsigned)g << 8 | (unsigned)gh[p] << 16 | (unsigned)hq << 24;
      ++hq;
      x = (x + 1) & 7;
    }
  static_assert((NOWN_U + GOWN - 1) / GOWN <= YMAX, "tile buffer too small");
  hipLaunchKernelGGL(ag_gemm_kernel, dim3(256), dim3(256), 0, stream, a);
  HIP_CHECK(hipGetLastError());
}
```

```cpp
#include <hip/hip_runtime.h>
#include <rccl/rccl.h>
#include <algorithm>
#include <cstdio>
#include <type_traits>
#include <vector>

#define HIP_CHECK(expr) do { hipError_t _e = (expr); if (_e != hipSuccess) { \
    fprintf(stderr, "HIP error %s at %s:%d\n", hipGetErrorString(_e), __FILE__, __LINE__); \
    return; } } while (0)

namespace {

#define CFG_KU 23
#define CFG_DEPTH 2
#define CFG_CLAIM_AT 8
#define CFG_TOKEN_AT 7
#define CFG_NDIR 3
#define CFG_DS0 7
#define CFG_DS1 4
#define CFG_PWIN 12
#define CFG_DELAY 48
#define CFG_STAGGER 12
#define CFG_NLOOP 24
#define CFG_NPRE 4
#define CFG_PSLEEP 2
#define CFG_WSLEEP 8

#define CFG_LPACE 4
#define NOB_OFF 0

constexpr int NRANK = 8;
constexpr int KDIM = 4096, MSH = 512, NSH = 512;
constexpr int KU = CFG_KU, GU = 32 - KU;
#define CFG_GOWN 5
#define CFG_GH {6, 6, 7, 9}
constexpr int GOWN = CFG_GOWN, NQUAD = 48, NHQ = NQUAD - 4 * GOWN;
constexpr int NOWN_U = 32 + 3 * KU + 4 * GU, NHLP_U = 4 * KU + 4 * GU;
constexpr int UNIT_B = 16 * KDIM * 2, ITEM_B = 16384;
constexpr int NDIR = CFG_NDIR, NST = 4 * (NHLP_U - GU - 3 * NDIR), NIT_Y = (NST + 7) / 8;
constexpr int NIT_IN = 8 * KU, NIT_X = 64 + 8 * GU, NIT_C = NIT_X + NIT_Y;
static_assert(NIT_Y <= 64, "one done flag per lane");

constexpr int XA_OFF = 0, XA_PAR = NRANK * 32 * UNIT_B;
constexpr int WB_OFF = XA_OFF + 2 * XA_PAR, WB_PAR = 4 * 4 * 16 * ITEM_B;
constexpr int YS_OFF = WB_OFF + 2 * WB_PAR, YS_LINK = 64 * ITEM_B;
constexpr int YD_OFF = YS_OFF + 4 * YS_LINK, YD_LINK = 64 * 128;
constexpr int XF_OFF = YD_OFF + 4 * YD_LINK, XF_PAR = NRANK * 32 * 128;
constexpr int WF_OFF = XF_OFF + 2 * XF_PAR, WF_PAR = 16 * 128;
constexpr int YF_OFF = WF_OFF + 2 * WF_PAR;
constexpr int YF_SRC = 1024;
constexpr int CNT_OFF = YF_OFF + 4 * YF_SRC;
constexpr int TMO_OFF = CNT_OFF + 1024 * 4;
constexpr int QC_OFF = TMO_OFF + 1024, YQ_OFF = QC_OFF + 1024;
constexpr int DBG_OFF = TMO_OFF + 4096;
constexpr int WS_USED = DBG_OFF + 1024 * 64 * 8;

constexpr int X_BYTES = MSH * KDIM * 4, W_BYTES = KDIM * NSH * 4, OUT_BYTES = NRANK * MSH * NSH * 4;
static_assert(X_BYTES == W_BYTES, "one descriptor size serves both inputs");
constexpr int RSRC_W3 = 0x00020000;
constexpr int AUX_SYS = 17;
constexpr int OOB = 0x7ffffff0;
constexpr unsigned long long DEADLINE_TICKS = 50000000ull;

typedef unsigned v4u __attribute__((ext_vector_type(4)));
typedef unsigned v2u __attribute__((ext_vector_type(2)));
typedef float f32x4 __attribute__((ext_vector_type(4)));
typedef float f32x2 __attribute__((ext_vector_type(2)));
typedef _Float16 h8 __attribute__((ext_vector_type(8)));
typedef _Float16 h2 __attribute__((ext_vector_type(2)));
typedef __amdgpu_buffer_rsrc_t rsrc_t;

struct Args {
  const float* x;
  const float* w;
  char* ws[NRANK];
  float* out[NRANK];
  int rank;
  unsigned quad[NQUAD];
  unsigned char list[2][160];
};

#define LAS __attribute__((address_space(3)))
#define SCHED_FENCE() __builtin_amdgcn_sched_barrier(0)
#define STR2(x) #x
#define STR(x) STR2(x)
#define WAIT_VM(n) asm volatile("s_waitcnt vmcnt(" STR(n) ")" ::: "memory")
#define WAIT_LGKM0() asm volatile("s_waitcnt lgkmcnt(0)" ::: "memory")

__device__ __forceinline__ rsrc_t mk_rsrc(const void* p, int bytes) {
  return __builtin_amdgcn_make_buffer_rsrc((void*)p, 0, bytes, RSRC_W3);
}
__device__ __forceinline__ unsigned rfl(unsigned v) { return (unsigned)__builtin_amdgcn_readfirstlane((int)v); }
__device__ __forceinline__ unsigned pk_f16(float a, float b) {
  h2 v = {(_Float16)a, (_Float16)b};
  return __builtin_bit_cast(unsigned, v);
}
__device__ __forceinline__ f32x4 mfma(v4u a, v4u b, f32x4 c) {
  return __builtin_amdgcn_mfma_f32_16x16x32_f16(__builtin_bit_cast(h8, a), __builtin_bit_cast(h8, b), c, 0, 0, 0);
}
template <int CTRL, int ROWMASK>
__device__ __forceinline__ unsigned dpp_max(unsigned m) {
  const unsigned o = (unsigned)__builtin_amdgcn_update_dpp((int)m, (int)m, CTRL, ROWMASK, 0xf, false);
  return o > m ? o : m;
}

__device__ __forceinline__ int block_scale(const v4u (&f)[32]) {
  float m = 0.f;
#pragma unroll
  for (int i = 0; i < 32; ++i) {
    const f32x4 v = __builtin_bit_cast(f32x4, f[i]);
    m = fmaxf(m, fmaxf(fmaxf(fabsf(v.x), fabsf(v.y)), fmaxf(fabsf(v.z), fabsf(v.w))));
  }
  unsigned u = __builtin_bit_cast(unsigned, m);
  u = dpp_max<0xB1, 0xf>(u);
  u = dpp_max<0x4E, 0xf>(u);
  u = dpp_max<0x141, 0xf>(u);
  u = dpp_max<0x128, 0xf>(u);
  const unsigned m0 = (unsigned)__builtin_amdgcn_readlane((int)u, 0), m1 = (unsigned)__builtin_amdgcn_readlane((int)u, 16);
  const unsigned m2 = (unsigned)__builtin_amdgcn_readlane((int)u, 32), m3 = (unsigned)__builtin_amdgcn_readlane((int)u, 48);
  const unsigned mm = max(max(m0, m1), max(m2, m3));
  int se = 127 + 14 + 127 - (int)(mm >> 23);
  return se > 254 ? 254 : se;
}

template <int PACE = 0>
__device__ __forceinline__ void load_block(v4u (&f)[32], rsrc_t rs, int vo, int vd, int so, int s0, int s1) {
#pragma unroll
  for (int m = 0; m < 16; ++m) {
    f[2 * m] = __builtin_amdgcn_raw_buffer_load_b128(rs, vo, so + (m >> 2) * s1 + (m & 3) * s0, 0);
    f[2 * m + 1] = __builtin_amdgcn_raw_buffer_load_b128(rs, vo + vd, so + (m >> 2) * s1 + (m & 3) * s0, 0);
    if (PACE > 0) {
      SCHED_FENCE();
      __builtin_amdgcn_s_sleep(PACE);
      SCHED_FENCE();
    }
  }
  SCHED_FENCE();
}
constexpr int X_VD = 16, X_S0 = KDIM * 4, X_S1 = 4 * KDIM * 4;
__device__ __forceinline__ int x_vo(int lane) { return lane * 32; }
__device__ __forceinline__ int x_so(int unit, int seg) { return unit * (16 * KDIM * 4) + seg * 2048; }
constexpr int W_VD = NSH * 4, W_S0 = 2 * NSH * 4, W_S1 = 64 * NSH * 4;
__device__ __forceinline__ int w_vo(int lane) { return (((lane >> 3) & 1) * 32 + (lane >> 4) * 8) * (NSH * 4) + (lane & 7) * 16; }
__device__ __forceinline__ int w_so(int chunk, int col0) { return chunk * (256 * NSH * 4) + col0 * 4; }

__device__ __forceinline__ int cvt_x(const v4u (&f)[32], v4u (&h)[16], char* lds, int lane) {
  const int se = block_scale(f);
  const float sc = __builtin_bit_cast(float, (unsigned)se << 23);
#pragma unroll
  for (int r = 0; r < 16; ++r) {
    const f32x4 a = __builtin_bit_cast(f32x4, f[2 * r]) * sc, b = __builtin_bit_cast(f32x4, f[2 * r + 1]) * sc;
    const v4u p = {pk_f16(a.x, a.y), pk_f16(a.z, a.w), pk_f16(b.x, b.y), pk_f16(b.z, b.w)};
    *(v4u*)(lds + lane * 256 + ((r ^ (lane & 15)) << 4)) = p;
  }
  const int q = lane >> 4, r = lane & 15;
#pragma unroll
  for (int s = 0; s < 16; ++s) {
    const int l = 4 * s + q;
    h[s] = *(const v4u*)(lds + l * 256 + ((r ^ (l & 15)) << 4));
  }
  return se;
}

__device__ __forceinline__ int cvt_w(const v4u (&f)[32], v4u (&h)[16]) {
  const int se = block_scale(f);
  const float sc = __builtin_bit_cast(float, (unsigned)se << 23);
#pragma unroll
  for (int t = 0; t < 4; ++t) {
    v4u p[4];
#pragma unroll
    for (int jj = 0; jj < 4; ++jj) {
      const f32x4 a = __builtin_bit_cast(f32x4, f[8 * t + 2 * jj]) * sc, b = __builtin_bit_cast(f32x4, f[8 * t + 2 * jj + 1]) * sc;
      p[0][jj] = pk_f16(a.x, b.x);
      p[1][jj] = pk_f16(a.y, b.y);
      p[2][jj] = pk_f16(a.z, b.z);
      p[3][jj] = pk_f16(a.w, b.w);
    }
#pragma unroll
    for (int c = 0; c < 2; ++c)
#pragma unroll
      for (int jj = 0; jj < 4; ++jj) {
        h[4 * t + c][jj] = (unsigned)__builtin_amdgcn_update_dpp((int)p[c][jj], (int)p[c + 2][jj], 0x128, 0xf, 0xc, false);
        h[4 * t + 2 + c][jj] = (unsigned)__builtin_amdgcn_update_dpp((int)p[c + 2][jj], (int)p[c][jj], 0x128, 0xf, 0x3, false);
      }
  }
  return se;
}

struct Waiter {
  rsrc_t rs;
  unsigned seq;
  unsigned long long t0;
  unsigned spins;
  bool dead;
  __device__ __forceinline__ bool give_up() {
    if (dead) return true;
    if ((++spins & 63u) != 0u) return false;
    const unsigned t = rfl(__builtin_amdgcn_raw_buffer_load_b32(rs, 0, TMO_OFF, AUX_SYS));
    if (t == seq || __builtin_amdgcn_s_memrealtime() - t0 > DEADLINE_TICKS) {
      __builtin_amdgcn_raw_buffer_store_b32(seq, rs, 0, TMO_OFF, AUX_SYS);
      dead = true;
      return true;
    }
    return false;
  }
};

#define STAMP(i) do { } while (0)
#define STAMPV(i, v) do { } while (0)
#define STAMP_OUT() do { } while (0)
#define STAMP_DECL() do { } while (0)

constexpr int RED_WAVE = 16 * 144, RED_BUF = 4 * RED_WAVE;
constexpr int LMAX = 32;
constexpr int LDS_CTL = 4 * 16384;
constexpr int LDS_BYTES = LDS_CTL + 128;
static_assert(2 * RED_BUF <= LDS_CTL, "reduction buffers reuse the transpose areas");

__global__ void __launch_bounds__(256, 1) ag_gemm_kernel(const Args a) {
  __shared__ __attribute__((aligned(16))) char lds[LDS_BYTES];
  const int lane = threadIdx.x & 63, w = (int)rfl(threadIdx.x >> 6);
  const int rank = a.rank, myidx = rank & 3, gbase = rank & 4, obase = gbase ^ 4;
  const int xcd = blockIdx.x & 7, slot = blockIdx.x >> 3;
  const rsrc_t rsS = mk_rsrc(a.ws[rank], WS_USED);
  const rsrc_t rsX = mk_rsrc(a.x, X_BYTES), rsW = mk_rsrc(a.w, W_BYTES);
  const int vone = lane == 0 ? 0 : OOB;
  char* const wlds = lds + w * 16384;
  LAS int* const token = (LAS int*)((LAS char*)lds + LDS_CTL);
  STAMP_DECL();
  STAMP(0);

  const int cnt_off = CNT_OFF + (blockIdx.x * 4 + w) * 4;
  const unsigned cnt_v = __builtin_amdgcn_raw_buffer_load_b32(rsS, 0, cnt_off, AUX_SYS);
  Waiter wt;
  wt.rs = rsS;
  wt.t0 = __builtin_amdgcn_s_memrealtime();
  wt.spins = 0;
  wt.dead = false;
#define TAKE_SEQ() \
  const unsigned seq = rfl(cnt_v) + 1u; \
  __builtin_amdgcn_raw_buffer_store_b32(seq, rsS, vone, cnt_off, AUX_SYS); \
  const int par = (int)(seq & 1u); \
  wt.seq = seq

  if (slot < 7) {
    const bool cross = slot >= 3;
    const unsigned peer = (unsigned)(cross ? obase + ((myidx + slot - 3) & 3) : gbase + ((myidx + 1 + slot) & 3));
    const int nit = cross ? NIT_C : NIT_IN, nld = cross ? NIT_X : NIT_IN;
    const rsrc_t rsD = mk_rsrc(a.ws[peer & 7u], WS_USED);
    v4u f[32], h[16];
    auto issue = [&](int n, auto pace) {
      const bool lw = cross && n < 64;
      const rsrc_t rs = mk_rsrc(lw ? a.w : a.x, X_BYTES);
      const int so = lw ? w_so(n & 15, (int)(peer & 3u) * 128 + (n >> 4) * 32) : x_so(((cross ? KU - 8 : 0) + (n >> 3)) & 31, n & 7);
      load_block<decltype(pace)::value>(f, rs, lw ? w_vo(lane) : x_vo(lane), lw ? W_VD : X_VD, so, lw ? W_S0 : X_S0, lw ? W_S1 : X_S1);
    };
    int vzero = 0;
    asm volatile("" : "+v"(vzero));
    int n = w * 8 + xcd;
    if (threadIdx.x == 0) *token = 0;
    WAIT_LGKM0();
    __builtin_amdgcn_s_barrier();
    for (int i = 0; i < w; ++i) __builtin_amdgcn_s_sleep(CFG_STAGGER);
    issue(n, std::integral_constant<int, 0>());
    TAKE_SEQ();
    const unsigned qbase = (seq - 1u) * (unsigned)nit - 32u;
    int turn = w;
    auto send = [&](rsrc_t rs, int vo, const int (&base)[8], int step, int foff, unsigned fval) __attribute__((always_inline)) {
      WAIT_LGKM0();
      if (turn < 4) STAMP(40);
      while ((int)rfl(__hip_atomic_load(token, __ATOMIC_RELAXED, __HIP_MEMORY_SCOPE_WORKGROUP)) < turn) __builtin_amdgcn_s_sleep(1);
      SCHED_FENCE();
      int claim = 0;
      const int deep = CFG_DS0 + ((nit - n) * CFG_DS1 >> 3);
#pragma unroll
      for (int s = 0; s < 16; ++s) {
        if (s == CFG_CLAIM_AT && lane == 0) claim = __builtin_amdgcn_raw_ptr_buffer_atomic_add_i32(1, rsS, vzero, QC_OFF + slot * 128, 0);
        __builtin_amdgcn_raw_buffer_store_b128(h[s], rs, vo, base[s >> 1] + (s & 1) * step, AUX_SYS);
        if (s == CFG_TOKEN_AT) __hip_atomic_fetch_max(token, turn + 1, __ATOMIC_RELAXED, __HIP_MEMORY_SCOPE_WORKGROUP);
        if (s < 15 && s < deep) WAIT_VM(CFG_DEPTH);
        SCHED_FENCE();
      }
      STAMPV(2 + 2 * ((turn >> 2) & 15), n);
      WAIT_VM(0);
      __builtin_amdgcn_raw_buffer_store_b32(fval, rsD, vone, foff, AUX_SYS);
      SCHED_FENCE();
      STAMP(3 + 2 * ((turn >> 2) & 15));
      const unsigned nn = rfl((unsigned)claim) - qbase;
      n = nn < (unsigned)nit ? (int)nn : nit;
      turn += 4;
    };
    for (;;) {
      const bool isw = cross && n < 64;
      const int unit = (cross ? KU - 8 : 0) + (n >> 3);
      int se;
      if (isw) se = cvt_w(f, h);
      else se = cvt_x(f, h, wlds, lane);
      const int doff = isw ? WB_OFF + par * WB_PAR + (myidx * 64 + n) * ITEM_B
                           : XA_OFF + par * XA_PAR + (rank * 32 + unit) * UNIT_B + (n & 7) * ITEM_B;
      const int foff = isw ? WF_OFF + par * WF_PAR + (myidx * 4 + (n >> 4)) * 128 + (n & 15) * 4
                           : XF_OFF + par * XF_PAR + (rank * 32 + unit) * 128 + (n & 7) * 4;
      const int base[8] = {doff, doff + 2048, doff + 4096, doff + 6144, doff + 8192, doff + 10240, doff + 12288, doff + 14336};
      send(rsD, lane * 16, base, 1024, foff, seq * 256u + (unsigned)se);
      if (n >= nld) break;
      issue(n, std::integral_constant<int, CFG_LPACE>());
    }
    const rsrc_t rsO = mk_rsrc(a.out[peer & 7u], OUT_BYTES);
    while (n < nit) {
      const int j = n - NIT_X;
      const bool need = lane < 32 && 8 * j + (lane >> 2) < NST;
      unsigned d;
      for (;;) {
        d = __builtin_amdgcn_raw_buffer_load_b32(rsS, need ? lane * 4 : OOB, YD_OFF + (slot - 3) * YD_LINK + j * 128, AUX_SYS);
        if (__all(!need || (d >> 12) == (seq & 0xfffffu))) break;
        if (wt.give_up()) break;
        __builtin_amdgcn_s_sleep(CFG_PSLEEP);
      }
#pragma unroll
      for (int s = 0; s < 16; ++s)
        h[s] = __builtin_amdgcn_raw_buffer_load_b128(rsS, lane * 16 + (s & 3) * 1024, YS_OFF + (slot - 3) * YS_LINK + j * ITEM_B + (s >> 2) * 4096, AUX_SYS);
      int base[8];
#pragma unroll
      for (int t = 0; t < 8; ++t) {
        const unsigned dt = (unsigned)__builtin_amdgcn_readlane((int)d, 4 * t);
        base[t] = (8 * j + t < NST && !wt.dead) ? (int)((dt >> 2) & 255u) * (16 * NSH * 4) + (myidx * 128 + (int)(dt & 3u) * 32) * 4 : OUT_BYTES;
      }
      WAIT_VM(0);
      send(rsO, (lane >> 3) * (NSH * 4) + (lane & 7) * 16, base, 8 * NSH * 4, YF_OFF + myidx * YF_SRC + 512 + j * 4, wt.dead ? 0u : seq);
    }
    __hip_atomic_fetch_max(token, 0x7fffffff, __ATOMIC_RELAXED, __HIP_MEMORY_SCOPE_WORKGROUP);
    STAMP_OUT();
    return;
  }

  if (slot == 7) {
    TAKE_SEQ();
    (void)par;
    if (xcd != 0) return;
    const int vo = lane < NHQ ? lane * 16 : OOB;
    for (;;) {
      const v4u v = __builtin_amdgcn_raw_buffer_load_b128(rsS, vo, YF_OFF + w * YF_SRC, AUX_SYS);
      const unsigned u = __builtin_amdgcn_raw_buffer_load_b32(rsS, lane * 4, YF_OFF + w * YF_SRC + 512, AUX_SYS);
      const bool ok = (lane >= NHQ || (v.x >= seq && v.y >= seq && v.z >= seq && v.w >= seq)) && (lane >= NIT_Y || u >= seq);
      if (__all(ok)) break;
      if (wt.give_up()) break;
      __builtin_amdgcn_s_sleep(2);
    }
    if (wt.dead) {
      const rsrc_t rsO = mk_rsrc(a.out[rank], OUT_BYTES);
      __builtin_amdgcn_raw_buffer_store_b32(0x7fc00000u, rsO, vone, rank * (MSH * NSH * 4), AUX_SYS);
    }
    STAMP(1);
    STAMP_OUT();
    return;
  }

  const int cj = slot - 8, hsrc = cj & 3;
  const unsigned qd = a.quad[xcd * 6 + (cj >> 2)];
  const bool helper = (qd & 4u) != 0u;
  const int hpan = (int)(qd & 3u), grp = (int)((qd >> 8) & 255u), ngrp = (int)((qd >> 16) & 255u), hq = (int)(qd >> 24);
  unsigned lst;
  int nun;
  {
    const int li = grp + lane * ngrp;
    const bool inr = li < (helper ? NHLP_U : NOWN_U);
    const unsigned e = a.list[helper ? 1 : 0][inr ? li : 0];
    const bool keep = inr && !(helper && (int)(e >> 5) == obase + hsrc);
    const unsigned long long km = __ballot(keep);
    nun = __builtin_popcountll(km);
    const int pos = keep ? (int)__builtin_amdgcn_mbcnt_hi((unsigned)(km >> 32), __builtin_amdgcn_mbcnt_lo((unsigned)km, 0u)) : 63;
    lst = (unsigned)__builtin_amdgcn_ds_permute(pos * 4, (int)e);
  }
  const unsigned dest = (unsigned)(helper ? obase + hsrc : rank);
  const int col0 = helper ? myidx * 128 + hpan * 32 : (hpan * 4 + hsrc) * 32;
  const rsrc_t rsO = mk_rsrc(a.out[dest & 7u], OUT_BYTES);

  v4u b[64];
  v4u av[32];
  int eb[4];
  TAKE_SEQ();
  __builtin_amdgcn_s_sleep(CFG_DELAY);
  if (helper) {
    const int hid = (hq * 4 + hsrc) * 4 + w;
    if (hid < 256) {
      v4u h[16];
      const int unit = hid >> 3, seg = hid & 7;
      load_block(av, rsX, x_vo(lane), X_VD, x_so(unit, seg), X_S0, X_S1);
      const int se = cvt_x(av, h, wlds, lane);
      const int doff = XA_OFF + par * XA_PAR + (rank * 32 + unit) * UNIT_B + seg * ITEM_B;
#pragma unroll
      for (int s = 0; s < 16; ++s)
        __builtin_amdgcn_raw_buffer_store_b128(h[s], rsS, lane * 16 + (s & 3) * 1024, doff + (s >> 2) * 4096, AUX_SYS);
      WAIT_VM(0);
      __builtin_amdgcn_raw_buffer_store_b32(seq * 256u + (unsigned)se, rsS, vone,
                                            XF_OFF + par * XF_PAR + (rank * 32 + unit) * 128 + seg * 4, AUX_SYS);
    }
    STAMP(1);
    const int fo = WF_OFF + par * WF_PAR + (hsrc * 4 + hpan) * 128 + w * 16;
    unsigned f0, f1, f2, f3;
    for (;;) {
      const v4u v = __builtin_amdgcn_raw_buffer_load_b128(rsS, 0, fo, AUX_SYS);
      f0 = rfl(v.x); f1 = rfl(v.y); f2 = rfl(v.z); f3 = rfl(v.w);
      if ((f0 >> 8) == seq && (f1 >> 8) == seq && (f2 >> 8) == seq && (f3 >> 8) == seq) break;
      if (wt.give_up()) break;
      __builtin_amdgcn_s_sleep(CFG_WSLEEP);
    }
    eb[0] = f0 & 255; eb[1] = f1 & 255; eb[2] = f2 & 255; eb[3] = f3 & 255;
    STAMP(2);
    const int bo = WB_OFF + par * WB_PAR + ((hsrc * 4 + hpan) * 16 + w * 4) * ITEM_B;
#pragma unroll
    for (int s = 0; s < 64; ++s)
      b[s] = __builtin_amdgcn_raw_buffer_load_b128(rsS, NOB_OFF + lane * 16 + (s & 3) * 1024, bo + (s >> 2) * 4096, AUX_SYS);
    SCHED_FENCE();
  } else {
    v4u fb[32], h[16];
    auto ldw = [&](v4u (&f)[32], int j) { load_block(f, rsW, NOB_OFF + w_vo(lane), W_VD, w_so(w * 4 + j, col0), W_S0, W_S1); };
    auto cvw = [&](const v4u (&f)[32], auto jt) {
      constexpr int j = decltype(jt)::value;
      eb[j] = cvt_w(f, h);
#pragma unroll
      for (int s = 0; s < 16; ++s) b[16 * j + s] = h[s];
      SCHED_FENCE();
    };
    ldw(av, 0);
    ldw(fb, 1);
    cvw(av, std::integral_constant<int, 0>());
    ldw(av, 2);
    cvw(fb, std::integral_constant<int, 1>());
    ldw(fb, 3);
    cvw(av, std::integral_constant<int, 2>());
    cvw(fb, std::integral_constant<int, 3>());
  }
  __syncthreads();
  STAMP(3);

  const int xa = XA_OFF + par * XA_PAR + w * (2 * ITEM_B);
  const int va = lane * 16;
  auto load_a = [&](int s, int e) {
    return __builtin_amdgcn_raw_buffer_load_b128(rsS, va + (s & 3) * 1024, xa + e * UNIT_B + (s >> 2) * 4096, AUX_SYS);
  };
  const int vflag = XF_OFF + par * XF_PAR + w * 8 + (int)lst * 128;
  const unsigned all = nun >= 32 ? 0xffffffffu : (1u << nun) - 1u;
  unsigned todo = all, watch = all;
  unsigned rdy = 0, part = 0, mine = 0;
  unsigned exps = 0;
  v2u pv;
  auto poll_issue = [&]() {
    const int lo = __builtin_ctz(watch | 0x80000000u);
    const bool on = lane >= lo && lane < lo + CFG_PWIN && ((watch & ~mine) >> (lane & 31) & 1u) != 0u;
    pv = __builtin_amdgcn_raw_buffer_load_b64(rsS, on ? vflag : OOB, 0, AUX_SYS);
  };
  auto poll_eval = [&]() {
    const bool ok = (pv.x >> 8) == seq && (pv.y >> 8) == seq;
    exps = ok ? (pv.x & 255u) | (pv.y & 255u) << 8 : exps;
    mine |= (unsigned)__ballot(ok);
    if (wt.dead) mine = 0xffffffffu;
  };
  LAS unsigned* const ctl = (LAS unsigned*)((LAS char*)lds + LDS_CTL);
  int rnd = 0;

  const int red_w = (lane & 15) * 144 + ((lane >> 4) & 1) * 64 + (lane >> 5) * 8;
  const int red_r = (w * 4 + ((lane >> 3) & 3)) * 144 + (lane & 7) * 16;
  const int vy = lane < 32 ? (w * 4 + (lane >> 3)) * (NSH * 4) + (lane & 7) * 16 : OOB;
  const int vs = lane < 32 ? w * 512 + lane * 16 : OOB;
  const int ylink = (hsrc - myidx) & 3;
  const unsigned ybase = (seq - 1u) * (unsigned)NST;
  int vzero = 0;
  asm volatile("" : "+v"(vzero));
  int pend_vo = OOB, pend_so = 0;
  unsigned pend_val = 0;
  auto emit = [&]() {
    __builtin_amdgcn_raw_buffer_store_b32(pend_val, rsS, pend_vo, pend_so, AUX_SYS);
    pend_vo = OOB;
  };
  auto choose = [&]() {
    unsigned c = rdy & todo;
    if (c == 0u) c = part & todo;
    if (c == 0u && (todo & (todo - 1u)) == 0u) c = todo;
    c &= 0u - c;
    todo &= ~c;
    return c;
  };
  auto share = [&](v4u m) {
    rdy = rfl(m.x & m.y & m.z & m.w);
    part = rfl(m.x | m.y | m.z | m.w);
    rnd ^= 1;
  };
  auto repoll = [&]() {
    wt.give_up();
    __builtin_amdgcn_s_sleep(CFG_PSLEEP);
    poll_issue();
    WAIT_VM(1);
    emit();
  };
  auto take = [&](unsigned bit, int& e, unsigned& ex2) {
    const int i = __builtin_ctz(bit);
    watch &= ~bit;
    e = __builtin_amdgcn_readlane((int)lst, i);
    ex2 = (unsigned)__builtin_amdgcn_readlane((int)exps, i);
  };

  auto unit_body = [&](auto pf_tag, int ecur, int enext, unsigned ex2) __attribute__((always_inline)) {
    constexpr bool PF = decltype(pf_tag)::value;
    constexpr int NLOOP = CFG_NLOOP, NPRE = CFG_NPRE;
    const int ea0 = (int)(ex2 & 255u), ea1 = (int)(ex2 >> 8);
    const int ex[4] = {254 - ea0 - eb[0], 254 - ea0 - eb[1], 254 - ea1 - eb[2], 254 - ea1 - eb[3]};
    const bool staged = helper && !((ecur >> 7) == (rank >> 2) && (ecur >> 5) != rank && (ecur & 31) >= KU - NDIR);
    int tslot = 0;
    if (staged && threadIdx.x == 0) tslot = __builtin_amdgcn_raw_ptr_buffer_atomic_add_i32(1, rsS, vzero, YQ_OFF + ylink * 128, 0);
    if (PF) poll_issue();
    SCHED_FENCE();
    const float bad = wt.dead ? __builtin_nanf("") : 0.f;
    f32x4 acc[4][2];
    f32x2 p[4];
#pragma unroll
    for (int j = 0; j < 4; ++j) {
      acc[j][0] = acc[j][1] = f32x4{0.f, 0.f, 0.f, 0.f};
      p[j] = f32x2{bad, bad};
    }
#pragma unroll
    for (int s = 0; s < 32; ++s) {
      const v4u cur = av[s];
      acc[s >> 3][0] = mfma(b[2 * s], cur, acc[s >> 3][0]);
      acc[s >> 3][1] = mfma(b[2 * s + 1], cur, acc[s >> 3][1]);
      if (PF && (s + 1) * NLOOP / 32 > s * NLOOP / 32) av[s * NLOOP / 32] = load_a(s * NLOOP / 32, enext);
      if (s >= 8 && (s & 7) >= 2 && (s & 7) < 6) {
        const int ch = (s >> 3) - 1, j = (s & 7) - 2;
        p[j][0] += __builtin_amdgcn_ldexpf(acc[ch][0][j], ex[ch]);
        p[j][1] += __builtin_amdgcn_ldexpf(acc[ch][1][j], ex[ch]);
        asm volatile("" : "+v"(p[j]));
      }
      SCHED_FENCE();
    }
    if (PF) {
      WAIT_VM(CFG_NLOOP);
      poll_eval();
    } else {
      WAIT_VM(0);
      STAMP(47);
      poll_issue();
    }
    emit();
    SCHED_FENCE();
    char* const rb = lds + rnd * RED_BUF;
#pragma unroll
    for (int j = 0; j < 4; ++j) {
      if (PF && j < NPRE) av[NLOOP + j] = load_a(NLOOP + j, enext);
      SCHED_FENCE();
      p[j][0] += __builtin_amdgcn_ldexpf(acc[3][0][j], ex[3]);
      p[j][1] += __builtin_amdgcn_ldexpf(acc[3][1][j], ex[3]);
      *(f32x2*)(rb + w * RED_WAVE + red_w + j * 16) = p[j];
      SCHED_FENCE();
    }
    ctl[rnd * 8 + w] = mine;
    if (threadIdx.x == 0) ctl[rnd * 8 + 4] = (unsigned)tslot - ybase;
    WAIT_LGKM0();
    __builtin_amdgcn_s_barrier();
    SCHED_FENCE();
    const f32x4 p0 = *(const f32x4*)(rb + red_r), p1 = *(const f32x4*)(rb + RED_WAVE + red_r);
    const f32x4 p2 = *(const f32x4*)(rb + 2 * RED_WAVE + red_r), p3 = *(const f32x4*)(rb + 3 * RED_WAVE + red_r);
    const v4u m = *(const LAS v4u*)(ctl + rnd * 8);
    const unsigned tsv = ctl[rnd * 8 + 4];
    SCHED_FENCE();
    if (PF) {
#pragma unroll
      for (int s = NLOOP + NPRE; s < 32; ++s) av[s] = load_a(s, enext);
    }
    SCHED_FENCE();
    f32x4 y = (p0 + p1) + (p2 + p3);
    y.x = y.x <= 0.f ? 0.f : y.x;
    y.y = y.y <= 0.f ? 0.f : y.y;
    y.z = y.z <= 0.f ? 0.f : y.z;
    y.w = y.w <= 0.f ? 0.f : y.w;
    share(m);
    const int ts = (int)(rfl(tsv) & 511u);
    if (staged) {
      __builtin_amdgcn_raw_buffer_store_b128(__builtin_bit_cast(v4u, y), rsS, vs, YS_OFF + ylink * YS_LINK + ts * 2048, AUX_SYS);
      pend_vo = vone;
      pend_so = YD_OFF + ylink * YD_LINK + ts * 16 + w * 4;
      pend_val = seq << 12 | (unsigned)ecur << 2 | (unsigned)hpan;
    } else {
      __builtin_amdgcn_raw_buffer_store_b128(__builtin_bit_cast(v4u, y), rsO, vy, ecur * (16 * NSH * 4) + col0 * 4, AUX_SYS);
    }
    SCHED_FENCE();
  };

  int ecur, enext;
  unsigned xcur, xnext, cb = 0;
  int cnt = 0;
  poll_issue();
  emit();
  for (;;) {
    while (cb == 0u) {
      poll_eval();
      ctl[rnd * 8 + w] = mine;
      WAIT_LGKM0();
      __builtin_amdgcn_s_barrier();
      share(*(const LAS v4u*)(ctl + rnd * 8));
      cb = choose();
      if ((cb & mine) == 0u) repoll();
    }
    while ((cb & mine) == 0u) {
      poll_eval();
      if ((cb & mine) == 0u) repoll();
    }
    take(cb, ecur, xcur);
    STAMP(44);
    STAMPV(46, ecur);
#pragma unroll
    for (int s = 0; s < 32; ++s) {
      av[s] = load_a(s, ecur);
      SCHED_FENCE();
    }
    STAMP(45);
    cb = choose();
    while (cb & mine) {
      take(cb, enext, xnext);
      unit_body(std::true_type(), ecur, enext, xcur);
      STAMP(4 + cnt);
      ++cnt;
      ecur = enext;
      xcur = xnext;
      cb = choose();
    }
    unit_body(std::false_type(), ecur, 0, xcur);
    STAMP(4 + cnt);
    ++cnt;
    if (watch == 0u) break;
  }
  (void)cnt;
  WAIT_VM(0);
  emit();
  if (helper) {
    const rsrc_t rsD = mk_rsrc(a.ws[dest & 7u], WS_USED);
    __builtin_amdgcn_raw_buffer_store_b32(seq, rsD, vone, YF_OFF + myidx * YF_SRC + (hq * 4 + w) * 4, AUX_SYS);
  }
  STAMP(40);
  STAMPV(41, (int)qd);
  STAMP_OUT();
}

struct Ev { double t; int e; };

}

extern "C" void kernel_launch(int rank, int nranks, ncclComm_t comm,
                              hipStream_t stream,
                              void* const* d_in, const int* in_sizes, int n_in,
                              void* d_out, int out_size,
                              const int* in_dtypes, int out_dtype,
                              void* const* d_out_all,
                              void* const* d_ws_all, size_t ws_bytes) {
  (void)comm; (void)in_dtypes; (void)out_dtype; (void)d_out;
  if (nranks != NRANK || n_in < 2 || (unsigned)rank >= (unsigned)NRANK || ws_bytes < (size_t)WS_USED ||
      in_sizes[0] != MSH * KDIM || in_sizes[1] != KDIM * NSH || out_size != NRANK * MSH * NSH) {
    fprintf(stderr, "unexpected problem shape\n");
    return;
  }
  Args a = {};
  a.x = (const float*)d_in[0];
  a.w = (const float*)d_in[1];
  for (int p = 0; p < NRANK; ++p) {
    a.ws[p] = (char*)d_ws_all[p];
    a.out[p] = (float*)d_out_all[p];
  }
  a.rank = rank;

  const int gbase = rank & 4, obase = gbase ^ 4;
  std::vector<Ev> ev;
  for (int u = 0; u < 32; ++u) ev.push_back({0.001 * u, rank * 32 + u});
  for (int p = gbase; p < gbase + 4; ++p)
    if (p != rank)
      for (int u = 0; u < KU; ++u) ev.push_back({1.0 + u + 0.01 * p, p * 32 + u});
  for (int p = obase; p < obase + 4; ++p)
    for (int v = 0; v < GU; ++v) ev.push_back({9.1 + v + 0.01 * p, p * 32 + KU + v});
  std::stable_sort(ev.begin(), ev.end(), [](const Ev& l, const Ev& r) { return l.t < r.t; });
  int n0 = 0, n1 = 0;
  for (const Ev& v : ev) {
    a.list[0][n0++] = (unsigned char)v.e;
    if (!((v.e >> 5) == rank && (v.e & 31) >= KU)) a.list[1][n1++] = (unsigned char)v.e;
  }
  const int gh[4] = CFG_GH;
  static_assert(NHQ <= 32 && NHQ >= 16, "done flags of one helper fit 512 bytes; the helpers convert 256 blocks");
  if (gh[0] + gh[1] + gh[2] + gh[3] != NHQ || n0 != NOWN_U || n1 != NHLP_U) { fprintf(stderr, "bad quad split\n"); return; }
  int used[8] = {0, 0, 0, 0, 0, 0, 0, 0};
  for (int c = 0; c < 4; ++c)
    for (int g = 0; g < GOWN; ++g) {
      const int x = c + 4 * (g & 1);
      a.quad[x * 6 + used[x]++] = (unsigned)c | (unsigned)g << 8 | (unsigned)GOWN << 16;
    }
  int hq = 0, x = 0;
  for (int p = 3; p >= 0; --p)
    for (int g = 0; g < gh[p]; ++g) {
      if ((NHLP_U + gh[p] - 1) / gh[p] > LMAX) { fprintf(stderr, "bad quad split\n"); return; }
      while (used[x] >= 6) x = (x + 1) & 7;
      a.quad[x * 6 + used[x]++] = (unsigned)(4 + p) | (unsigned)g << 8 | (unsigned)gh[p] << 16 | (unsigned)hq << 24;
      ++hq;
      x = (x + 1) & 7;
    }
  static_assert((NOWN_U + GOWN - 1) / GOWN <= LMAX, "one list entry per bit");
  hipLaunchKernelGGL(ag_gemm_kernel, dim3(256), dim3(256), 0, stream, a);
  HIP_CHECK(hipGetLastError());
}
```

```cpp
#include <hip/hip_runtime.h>
#include <rccl/rccl.h>
#include <algorithm>
#include <cstdio>
#include <type_traits>
#include <vector>

#define HIP_CHECK(expr) do { hipError_t _e = (expr); if (_e != hipSuccess) { \
    fprintf(stderr, "HIP error %s at %s:%d\n", hipGetErrorString(_e), __FILE__, __LINE__); \
    return; } } while (0)

namespace {

#define CFG_KU 22
#define CFG_DEPTH 2
#define CFG_DEPTH0 5
#define CFG_CLAIM_AT 8
#define CFG_TOKEN_AT 7
#define CFG_NDIR 4
#define CFG_DS0 7
#define CFG_DS1 4
#define CFG_NAGPR 16
#define CFG_EQ 7
#define CFG_DELAY 48
#define CFG_STAGGER 12
#define CFG_NLOOP 20
#define CFG_NPRE 4
#define CFG_POLL_AT 12
#define CFG_DS0Y 4
#define CFG_DS1Y 4
#define CFG_PSLEEP 1
#define CFG_WSLEEP 8

#define CFG_LPACE 4

constexpr int NRANK = 8;
constexpr int KDIM = 4096, MSH = 512, NSH = 512;
constexpr int KU = CFG_KU, GU = 32 - KU;
#define CFG_NXQ 2
#define CFG_NPP 40
#define CFG_NRAW 6
#define CFG_DEPTHC 3
#define CFG_GOWN 5
#define CFG_GH {6, 6, 6, 8}
constexpr int GOWN = CFG_GOWN, NQUAD = 48, NXQ = CFG_NXQ, NHQ = NQUAD - 4 * GOWN - NXQ;
constexpr int NXP = 8 + 4 * NXQ;
constexpr int NOWN_U = 32 + 3 * KU + 4 * GU, NHLP_U = 4 * KU + 4 * GU;
constexpr int UNIT_B = 16 * KDIM * 2, ITEM_B = 16384;
constexpr int NDIR = CFG_NDIR, NST = 4 * (NHLP_U - GU - 3 * NDIR), NIT_Y = (NST + 7) / 8;
constexpr int NIT_IN = 8 * KU, NIT_X = 64 + 8 * GU, NIT_C = NIT_X + NIT_Y;
static_assert(NIT_Y <= 64 && CFG_NRAW >= 1 && CFG_NPP <= NIT_Y - CFG_NRAW, "the last, possibly short block travels unpacked");

constexpr int XA_OFF = 0, XA_PAR = NRANK * 32 * UNIT_B;
constexpr int WB_OFF = XA_OFF + 2 * XA_PAR, WB_PAR = 32 * 16 * ITEM_B;
constexpr int YS_OFF = WB_OFF + 2 * WB_PAR, YS_LINK = 64 * ITEM_B;
constexpr int YC_OFF = YS_OFF + 4 * YS_LINK;
constexpr int YP_OFF = YC_OFF + 4 * YS_LINK;
constexpr int YD_OFF = YP_OFF + 4 * YS_LINK, YD_LINK = 64 * 128;
constexpr int XF_OFF = YD_OFF + 4 * YD_LINK, XF_PAR = NRANK * 32 * 128;
constexpr int WF_OFF = XF_OFF + 2 * XF_PAR, WF_PAR = 32 * 128;
constexpr int YF_OFF = WF_OFF + 2 * WF_PAR;
constexpr int YF_SRC = 1024;
constexpr int CNT_OFF = YF_OFF + 4 * YF_SRC;
constexpr int TMO_OFF = CNT_OFF + 1024 * 4;
constexpr int QC_OFF = TMO_OFF + 1024, YQ_OFF = QC_OFF + 1024, PF_OFF = YQ_OFF + 1024;
constexpr int DBG_OFF = TMO_OFF + 4096;
constexpr int WS_USED = DBG_OFF + 1024 * 64 * 8;

constexpr int X_BYTES = MSH * KDIM * 4, W_BYTES = KDIM * NSH * 4, OUT_BYTES = NRANK * MSH * NSH * 4;
static_assert(X_BYTES == W_BYTES, "one descriptor size serves both inputs");
constexpr int RSRC_W3 = 0x00020000;
constexpr int AUX_SYS = 17;
constexpr int OOB = 0x7ffffff0;
constexpr unsigned long long DEADLINE_TICKS = 50000000ull;

typedef unsigned v4u __attribute__((ext_vector_type(4)));
typedef unsigned v2u __attribute__((ext_vector_type(2)));
typedef float f32x4 __attribute__((ext_vector_type(4)));
typedef float f32x2 __attribute__((ext_vector_type(2)));
typedef _Float16 h8 __attribute__((ext_vector_type(8)));
typedef _Float16 h2 __attribute__((ext_vector_type(2)));
typedef __amdgpu_buffer_rsrc_t rsrc_t;

struct Args {
  const float* x;
  const float* w;
  char* ws[NRANK];
  float* out[NRANK];
  int rank;
  unsigned quad[NQUAD];
  unsigned char list[2][160];
};

#define INL __attribute__((always_inline))
#define LAS __attribute__((address_space(3)))
#define SCHED_FENCE() __builtin_amdgcn_sched_barrier(0)
#define WAIT_VM(n) asm volatile("s_waitcnt vmcnt(%0)" ::"n"(n) : "memory")
#define WAIT_LGKM0() asm volatile("s_waitcnt lgkmcnt(0)" ::: "memory")

__device__ __forceinline__ rsrc_t mk_rsrc(const void* p, int bytes) {
  return __builtin_amdgcn_make_buffer_rsrc((void*)p, 0, bytes, RSRC_W3);
}
__device__ __forceinline__ unsigned rfl(unsigned v) { return (unsigned)__builtin_amdgcn_readfirstlane((int)v); }
__device__ __forceinline__ unsigned pk_f16(float a, float b) {
  h2 v = {(_Float16)a, (_Float16)b};
  return __builtin_bit_cast(unsigned, v);
}
__device__ __forceinline__ f32x4 mfma(v4u a, v4u b, f32x4 c) {
  return __builtin_amdgcn_mfma_f32_16x16x32_f16(__builtin_bit_cast(h8, a), __builtin_bit_cast(h8, b), c, 0, 0, 0);
}
template <int CTRL, int ROWMASK>
__device__ __forceinline__ unsigned dpp_max(unsigned m) {
  const unsigned o = (unsigned)__builtin_amdgcn_update_dpp((int)m, (int)m, CTRL, ROWMASK, 0xf, false);
  return o > m ? o : m;
}

__device__ __forceinline__ int block_scale(const v4u (&f)[32]) {
  float m = 0.f;
#pragma unroll
  for (int i = 0; i < 32; ++i) {
    asm volatile("v_max3_f32 %0, |%1|, |%2|, %0\n\tv_max3_f32 %0, |%3|, |%4|, %0"
                 : "+v"(m) : "v"(f[i].x), "v"(f[i].y), "v"(f[i].z), "v"(f[i].w));
  }
  unsigned u = __builtin_bit_cast(unsigned, m);
  u = dpp_max<0xB1, 0xf>(u);
  u = dpp_max<0x4E, 0xf>(u);
  u = dpp_max<0x141, 0xf>(u);
  u = dpp_max<0x128, 0xf>(u);
  const unsigned m0 = (unsigned)__builtin_amdgcn_readlane((int)u, 0), m1 = (unsigned)__builtin_amdgcn_readlane((int)u, 16);
  const unsigned m2 = (unsigned)__builtin_amdgcn_readlane((int)u, 32), m3 = (unsigned)__builtin_amdgcn_readlane((int)u, 48);
  const unsigned mm = max(max(m0, m1), max(m2, m3));
  int se = 127 + 14 + 127 - (int)((mm >> 23) | CFG_EQ);
  return se > 254 ? 254 : se;
}

template <int PACE = 0>
__device__ __forceinline__ void load_block(v4u (&f)[32], rsrc_t rs, int vo, int vd, int so, int s0, int s1) {
#pragma unroll
  for (int m = 0; m < 16; ++m) {
    f[2 * m] = __builtin_amdgcn_raw_buffer_load_b128(rs, vo, so + (m >> 2) * s1 + (m & 3) * s0, 0);
    f[2 * m + 1] = __builtin_amdgcn_raw_buffer_load_b128(rs, vo + vd, so + (m >> 2) * s1 + (m & 3) * s0, 0);
    SCHED_FENCE();
    if (PACE > 0) {
      __builtin_amdgcn_s_sleep(PACE);
      SCHED_FENCE();
    }
  }
  SCHED_FENCE();
}
constexpr int X_VD = 16, X_S0 = KDIM * 4, X_S1 = 4 * KDIM * 4;
__device__ __forceinline__ int x_vo(int lane) { return lane * 32; }
__device__ __forceinline__ int x_so(int unit, int seg) { return unit * (16 * KDIM * 4) + seg * 2048; }
constexpr int W_VD = NSH * 4, W_S0 = 2 * NSH * 4, W_S1 = 64 * NSH * 4;
__device__ __forceinline__ int w_vo(int lane) { return (((lane >> 3) & 1) * 32 + (lane >> 4) * 8) * (NSH * 4) + (lane & 7) * 16; }
__device__ __forceinline__ int w_so(int chunk, int col0) { return chunk * (256 * NSH * 4) + col0 * 4; }

__device__ __forceinline__ int cvt_x(const v4u (&f)[32], v4u (&h)[16], char* lds, int lane) {
  const int se = block_scale(f);
  const float sc = __builtin_bit_cast(float, (unsigned)se << 23);
#pragma unroll
  for (int r = 0; r < 16; ++r) {
    const f32x4 a = __builtin_bit_cast(f32x4, f[2 * r]) * sc, b = __builtin_bit_cast(f32x4, f[2 * r + 1]) * sc;
    const v4u p = {pk_f16(a.x, a.y), pk_f16(a.z, a.w), pk_f16(b.x, b.y), pk_f16(b.z, b.w)};
    *(v4u*)(lds + lane * 256 + ((r ^ (lane & 15)) << 4)) = p;
  }
  const int q = lane >> 4, r = lane & 15;
#pragma unroll
  for (int s = 0; s < 16; ++s) {
    const int l = 4 * s + q;
    h[s] = *(const v4u*)(lds + l * 256 + ((r ^ (l & 15)) << 4));
  }
  return se;
}

__device__ __forceinline__ int cvt_w(const v4u (&f)[32], v4u (&h)[16]) {
  const int se = block_scale(f);
  const float sc = __builtin_bit_cast(float, (unsigned)se << 23);
#pragma unroll
  for (int t = 0; t < 4; ++t) {
    v4u p[4];
#pragma unroll
    for (int jj = 0; jj < 4; ++jj) {
      const f32x4 a = __builtin_bit_cast(f32x4, f[8 * t + 2 * jj]) * sc, b = __builtin_bit_cast(f32x4, f[8 * t + 2 * jj + 1]) * sc;
      p[0][jj] = pk_f16(a.x, b.x);
      p[1][jj] = pk_f16(a.y, b.y);
      p[2][jj] = pk_f16(a.z, b.z);
      p[3][jj] = pk_f16(a.w, b.w);
    }
#pragma unroll
    for (int c = 0; c < 2; ++c)
#pragma unroll
      for (int jj = 0; jj < 4; ++jj) {
        h[4 * t + c][jj] = (unsigned)__builtin_amdgcn_update_dpp((int)p[c][jj], (int)p[c + 2][jj], 0x128, 0xf, 0xc, false);
        h[4 * t + 2 + c][jj] = (unsigned)__builtin_amdgcn_update_dpp((int)p[c + 2][jj], (int)p[c][jj], 0x128, 0xf, 0x3, false);
      }
  }
  return se;
}

constexpr int PK_DESC = 512, PK_DATA = 544, PK_MAX = 14;
template <int CTRL, int ROWMASK>
__device__ __forceinline__ int dpp_add(int v) { return v + __builtin_amdgcn_update_dpp(0, v, CTRL, ROWMASK, 0xf, false); }
__device__ __forceinline__ int wave_scan(int v) {
  v = dpp_add<0x111, 0xf>(v);
  v = dpp_add<0x112, 0xf>(v);
  v = dpp_add<0x114, 0xf>(v);
  v = dpp_add<0x118, 0xf>(v);
  v = dpp_add<0x142, 0xa>(v);
  v = dpp_add<0x143, 0xc>(v);
  return v;
}

struct Waiter {
  rsrc_t rs;
  unsigned seq;
  unsigned long long t0;
  unsigned spins;
  bool dead;
  __device__ __forceinline__ bool give_up() {
    if (dead) return true;
    if ((++spins & 63u) != 0u) return false;
    const unsigned t = rfl(__builtin_amdgcn_raw_buffer_load_b32(rs, 0, TMO_OFF, AUX_SYS));
    if (t == seq || __builtin_amdgcn_s_memrealtime() - t0 > DEADLINE_TICKS) {
      __builtin_amdgcn_raw_buffer_store_b32(seq, rs, 0, TMO_OFF, AUX_SYS);
      dead = true;
      return true;
    }
    return false;
  }
};

__device__ __forceinline__ unsigned fetch_block(v4u (&h)[16], Waiter& wt, int link, int j, int lane) {
  const bool need = lane < 32 && 8 * j + (lane >> 2) < NST;
  unsigned d;
  for (;;) {
    d = __builtin_amdgcn_raw_buffer_load_b32(wt.rs, need ? lane * 4 : +OOB, YD_OFF + link * YD_LINK + j * 128, AUX_SYS);
    if (__all(!need || (d >> 12) == (wt.seq & 0xfffffu))) break;
    if (wt.give_up()) break;
    __builtin_amdgcn_s_sleep(CFG_PSLEEP);
  }
#pragma unroll
  for (int s = 0; s < 16; ++s)
    h[s] = __builtin_amdgcn_raw_buffer_load_b128(wt.rs, lane * 16 + (s & 3) * 1024, YS_OFF + link * YS_LINK + j * ITEM_B + (s >> 2) * 4096, AUX_SYS);
  return d;
}
__device__ __forceinline__ int pack_block(const v4u (&h)[16], unsigned d, LAS char* pk, int lane) {
  unsigned mk[2] = {0u, 0u};
#pragma unroll
  for (int r = 0; r < 64; ++r) mk[r >> 5] |= (h[r >> 2][r & 3] != 0u ? 1u : 0u) << (r & 31);
  const int cnt = __builtin_popcount(mk[0]) + __builtin_popcount(mk[1]) + 1;
  const int inc = wave_scan(cnt);
  const int total = PK_DATA + 4 * __builtin_amdgcn_readlane(inc, 63);
  if (total > PK_MAX * 1024) return 0;
  *(LAS v2u*)(pk + lane * 8) = v2u{mk[0], mk[1]};
  if ((lane & 3) == 0 && lane < 32) *(LAS unsigned*)(pk + PK_DESC + lane) = d;
  int ptr = PK_DATA + 4 * (inc - cnt);
#pragma unroll
  for (int r = 0; r < 64; ++r) {
    *(LAS unsigned*)(pk + ptr) = h[r >> 2][r & 3];
    ptr += (int)(mk[r >> 5] >> (r & 31) & 1u) * 4;
  }
  return total;
}
__device__ __forceinline__ void load_packed(rsrc_t rs, int off, int np, LAS char* pk, int lane) {
  v4u t[PK_MAX];
#pragma unroll
  for (int p = 0; p < PK_MAX; ++p) t[p] = __builtin_amdgcn_raw_buffer_load_b128(rs, p < np ? lane * 16 : +OOB, off + p * 1024, AUX_SYS);
#pragma unroll
  for (int p = 0; p < PK_MAX; ++p) *(LAS v4u*)(pk + p * 1024 + lane * 16) = t[p];
}

#define STAMP(i) do { } while (0)
#define STAMPV(i, v) do { } while (0)
#define CYC(i) do { } while (0)
#define STAMP_OUT() do { } while (0)
#define STAMP_DECL() do { } while (0)

constexpr int RED_WAVE = 16 * 144, RED_BUF = 4 * RED_WAVE;
constexpr int LMAX = 31;
constexpr int LDS_CTL = 4 * 16384;
constexpr int LDS_BYTES = LDS_CTL + 128;
static_assert(RED_BUF + 32 <= 16384, "reduction buffers reuse the transpose areas");

__global__ void __launch_bounds__(256, 1) ag_gemm_kernel(const Args a) {
  __shared__ __attribute__((aligned(16))) char lds[LDS_BYTES];
  const int lane = threadIdx.x & 63, w = (int)rfl(threadIdx.x >> 6);
  const int rank = a.rank, myidx = rank & 3, gbase = rank & 4, obase = gbase ^ 4;
  const int xcd = blockIdx.x & 7, slot = blockIdx.x >> 3;
  const rsrc_t rsS = mk_rsrc(a.ws[rank], WS_USED);
  const int vone = lane == 0 ? 0 : OOB;
  char* const wlds = lds + w * 16384;
  LAS int* const token = (LAS int*)((LAS char*)lds + LDS_CTL);
  STAMP_DECL();
  STAMP(0);
  STAMPV(50, (int)(unsigned)__builtin_amdgcn_s_memtime());

  const int cnt_off = CNT_OFF + (blockIdx.x * 4 + w) * 4;
  const unsigned cnt_v = __builtin_amdgcn_raw_buffer_load_b32(rsS, 0, cnt_off, AUX_SYS);
  Waiter wt;
  wt.rs = rsS;
  wt.t0 = __builtin_amdgcn_s_memrealtime();
  wt.spins = 0;
  wt.dead = false;
#define TAKE_SEQ() \
  const unsigned seq = rfl(cnt_v) + 1u; \
  const int par = (int)(seq & 1u); \
  wt.seq = seq
#define PUT_SEQ() __builtin_amdgcn_raw_buffer_store_b32(seq, rsS, vone, cnt_off, AUX_SYS)

  if (slot < 7) {
    const bool cross = slot >= 3;
    const unsigned peer = (unsigned)(cross ? obase + ((myidx + slot - 3) & 3) : gbase + ((myidx + 1 + slot) & 3));
    const int nit = cross ? NIT_C : NIT_IN, nld = cross ? NIT_X : NIT_IN;
    const rsrc_t rsD = mk_rsrc(a.ws[peer & 7u], WS_USED);
    v4u f[32], h[16];
    auto issue = [&](int n, auto pace) INL {
      const bool lw = cross && n < 64;
      const rsrc_t rs = mk_rsrc(lw ? a.w : a.x, X_BYTES);
      const int so = lw ? w_so(n & 15, (int)(peer & 3u) * 128 + (n >> 4) * 32) : x_so(((cross ? KU - 8 : 0) + (n >> 3)) & 31, n & 7);
      load_block<decltype(pace)::value>(f, rs, lw ? w_vo(lane) : x_vo(lane), lw ? W_VD : X_VD, so, lw ? W_S0 : X_S0, lw ? W_S1 : X_S1);
    };
    int vzero = 0;
    asm volatile("" : "+v"(vzero));
    int n = w * 8 + xcd;
    if (threadIdx.x == 0) *token = 0;
    WAIT_LGKM0();
    __builtin_amdgcn_s_barrier();
    for (int i = 0; i < w; ++i) __builtin_amdgcn_s_sleep(CFG_STAGGER);
    issue(n, std::integral_constant<int, 0>());
    TAKE_SEQ();
    const unsigned qbase = (seq - 1u) * (unsigned)nit - 32u;
    int turn = w;
    auto send = [&](rsrc_t rs, int vo, const int (&base)[8], int step, int foff, unsigned fval) INL {
      WAIT_LGKM0();
      if (turn < 4) STAMP(40);
      while ((int)rfl(__hip_atomic_load(token, __ATOMIC_RELAXED, __HIP_MEMORY_SCOPE_WORKGROUP)) < turn) __builtin_amdgcn_s_sleep(1);
      SCHED_FENCE();
      int claim = 0;
      const int deep = cross ? CFG_DS0Y + ((nit - n) * CFG_DS1Y >> 3) : CFG_DS0 + ((nit - n) * CFG_DS1 >> 3);
#pragma unroll
      for (int s = 0; s < 16; ++s) {
        if (s == CFG_CLAIM_AT && lane == 0) claim = __builtin_amdgcn_raw_ptr_buffer_atomic_add_i32(1, rsS, vzero, QC_OFF + slot * 128, 0);
        __builtin_amdgcn_raw_buffer_store_b128(h[s], rs, vo, base[s >> 1] + (s & 1) * step, AUX_SYS);
        if (s == CFG_TOKEN_AT) __hip_atomic_fetch_max(token, turn + 1, __ATOMIC_RELAXED, __HIP_MEMORY_SCOPE_WORKGROUP);
        if (s < CFG_TOKEN_AT && turn == 0) WAIT_VM(CFG_DEPTH0);
        else if (s < 15 && s < deep) WAIT_VM(CFG_DEPTH);
        SCHED_FENCE();
      }
      STAMPV(2 + 2 * ((turn >> 2) & 15), n);
      WAIT_VM(0);
      __builtin_amdgcn_raw_buffer_store_b32(fval, rsD, vone, foff, AUX_SYS);
      SCHED_FENCE();
      STAMP(3 + 2 * ((turn >> 2) & 15));
      const unsigned nn = rfl((unsigned)claim) - qbase;
      n = nn < (unsigned)nit ? (int)nn : nit;
      turn += 4;
    };
    for (;;) {
      const bool isw = cross && n < 64;
      const int unit = (cross ? KU - 8 : 0) + (n >> 3);
      int se;
      if (isw) se = cvt_w(f, h);
      else se = cvt_x(f, h, wlds, lane);
      const int doff = isw ? WB_OFF + par * WB_PAR + (myidx * 64 + n) * ITEM_B
                           : XA_OFF + par * XA_PAR + (rank * 32 + unit) * UNIT_B + (n & 7) * ITEM_B;
      const int foff = isw ? WF_OFF + par * WF_PAR + (myidx * 4 + (n >> 4)) * 128 + (n & 15) * 4
                           : XF_OFF + par * XF_PAR + (rank * 32 + unit) * 128 + (n & 7) * 4;
      const int base[8] = {doff, doff + 2048, doff + 4096, doff + 6144, doff + 8192, doff + 10240, doff + 12288, doff + 14336};
      send(rsD, lane * 16, base, 1024, foff, seq * 256u + (unsigned)se);
      if (n >= nld) break;
      issue(n, std::integral_constant<int, CFG_LPACE>());
    }
    const rsrc_t rsO = mk_rsrc(a.out[peer & 7u], OUT_BYTES);
    unsigned pfv = 0;
    while (n < nit) {
      const int j = n - NIT_X;
      const int foff = YF_OFF + myidx * YF_SRC + 512 + j * 4;
      LAS char* const pk = (LAS char*)lds + w * 16384;
      int total = 0, claim = 0;
      if (j < CFG_NPP) {
        unsigned fl = (unsigned)__builtin_amdgcn_readlane((int)pfv, j);
        while ((fl >> 14) != (seq & 0x3ffffu)) {
          pfv = __builtin_amdgcn_raw_buffer_load_b32(rsS, lane * 4, PF_OFF + (slot - 3) * 256, AUX_SYS);
          fl = (unsigned)__builtin_amdgcn_readlane((int)pfv, j);
          if (wt.give_up()) break;
        }
        total = wt.dead ? 0 : (int)(fl & 0x3fffu);
        if (total) {
          if (lane == 0) claim = __builtin_amdgcn_raw_ptr_buffer_atomic_add_i32(1, rsS, vzero, QC_OFF + slot * 128, 0);
          load_packed(rsS, YP_OFF + (slot - 3) * YS_LINK + j * ITEM_B, (total + 1023) >> 10, pk, lane);
        }
      }
      unsigned d = 0;
      if (!total) {
        d = fetch_block(h, wt, slot - 3, j, lane);
        if (j >= CFG_NPP && j < NIT_Y - CFG_NRAW && !wt.dead) total = pack_block(h, d, pk, lane);
        if (total && lane == 0) claim = __builtin_amdgcn_raw_ptr_buffer_atomic_add_i32(1, rsS, vzero, QC_OFF + slot * 128, 0);
      }
      if (total) {
        const int np = (total + 1023) >> 10, doff = YC_OFF + myidx * YS_LINK + j * ITEM_B;
        while ((int)rfl(__hip_atomic_load(token, __ATOMIC_RELAXED, __HIP_MEMORY_SCOPE_WORKGROUP)) < turn) __builtin_amdgcn_s_sleep(1);
        for (int p = 0; p < np; ++p) {
          const v4u v = *(const LAS v4u*)(pk + p * 1024 + lane * 16);
          __builtin_amdgcn_raw_buffer_store_b128(v, rsD, p * 1024 + lane * 16 < total ? lane * 16 : +OOB, doff + p * 1024, AUX_SYS);
          if (p == (np - 1) / 2) __hip_atomic_fetch_max(token, turn + 1, __ATOMIC_RELAXED, __HIP_MEMORY_SCOPE_WORKGROUP);
          WAIT_VM(CFG_DEPTHC);
        }
        STAMPV(2 + 2 * ((turn >> 2) & 15), n);
        WAIT_VM(0);
        __builtin_amdgcn_raw_buffer_store_b32(seq << 5 | (unsigned)np, rsD, vone, foff, AUX_SYS);
        STAMP(3 + 2 * ((turn >> 2) & 15));
        const unsigned nn = rfl((unsigned)claim) - qbase;
        n = nn < (unsigned)nit ? (int)nn : nit;
        turn += 4;
        continue;
      }
      int base[8];
#pragma unroll
      for (int t = 0; t < 8; ++t) {
        const unsigned dt = (unsigned)__builtin_amdgcn_readlane((int)d, 4 * t);
        base[t] = (8 * j + t < NST && !wt.dead) ? (int)((dt >> 2) & 255u) * (16 * NSH * 4) + (myidx * 128 + (int)(dt & 3u) * 32) * 4 : OUT_BYTES;
      }
      WAIT_VM(0);
      send(rsO, (lane >> 3) * (NSH * 4) + (lane & 7) * 16, base, 8 * NSH * 4, foff, wt.dead ? 0u : seq << 5);
    }
    __hip_atomic_fetch_max(token, 0x7fffffff, __ATOMIC_RELAXED, __HIP_MEMORY_SCOPE_WORKGROUP);
    PUT_SEQ();
    STAMP_OUT();
    return;
  }

  const int cj = slot - 8, hsrc = cj & 3;
  const unsigned qd = slot == 7 ? 8u : a.quad[xcd * 6 + (cj >> 2)];
  if (qd & 8u) {
    TAKE_SEQ();
    (void)par;
    PUT_SEQ();
    const int xi = slot == 7 ? xcd : 8 + (int)(qd >> 24) * 4 + hsrc;
    const rsrc_t rsO = mk_rsrc(a.out[rank], OUT_BYTES);
    LAS char* const pk = (LAS char*)lds + w * 16384;
    for (int j = xi; j < CFG_NPP; j += NXP) {
      v4u h[16];
      const unsigned d = fetch_block(h, wt, w, j, lane);
      const int total = wt.dead ? 0 : pack_block(h, d, pk, lane);
      for (int p = 0; p * 1024 < total; ++p)
        __builtin_amdgcn_raw_buffer_store_b128(*(const LAS v4u*)(pk + p * 1024 + lane * 16), rsS, lane * 16,
                                               YP_OFF + w * YS_LINK + j * ITEM_B + p * 1024, AUX_SYS);
      WAIT_VM(0);
      __builtin_amdgcn_raw_buffer_store_b32(seq << 14 | (unsigned)total, rsS, vone, PF_OFF + w * 256 + j * 4, AUX_SYS);
      STAMP(10 + ((j / NXP) & 7));
    }
    for (int j = xi; j < NIT_Y; j += NXP) {
      unsigned fl;
      for (;;) {
        fl = rfl(__builtin_amdgcn_raw_buffer_load_b32(rsS, 0, YF_OFF + w * YF_SRC + 512 + j * 4, AUX_SYS));
        if ((fl >> 5) == (seq & 0x7ffffffu)) break;
        if (wt.give_up()) break;
        __builtin_amdgcn_s_sleep(2);
      }
      const int np = wt.dead ? 0 : (int)(fl & 31u);
      if (np == 0) continue;
      v4u h[16];
      load_packed(rsS, YC_OFF + w * YS_LINK + j * ITEM_B, np, pk, lane);
      const v2u mk = *(const LAS v2u*)(pk + lane * 8);
      const unsigned d = *(const LAS unsigned*)(pk + PK_DESC + (lane & 7) * 4);
      const int cnt = __builtin_popcount(mk.x) + __builtin_popcount(mk.y) + 1;
      int ptr = PK_DATA + 4 * (wave_scan(cnt) - cnt);
#pragma unroll
      for (int r = 0; r < 64; ++r) {
        const int m = (int)(mk[r >> 5] << (31 - (r & 31))) >> 31;
        h[r >> 2][r & 3] = *(const LAS unsigned*)(pk + (ptr & 16380)) & (unsigned)m;
        ptr -= 4 * m;
      }
#pragma unroll
      for (int t = 0; t < 8; ++t) {
        const unsigned dt = (unsigned)__builtin_amdgcn_readlane((int)d, t);
        const int so = (int)((dt >> 2) & 255u) * (16 * NSH * 4) + (w * 128 + (int)(dt & 3u) * 32) * 4;
        const int vo = (lane >> 3) * (NSH * 4) + (lane & 7) * 16;
        __builtin_amdgcn_raw_buffer_store_b128(h[2 * t], rsO, vo, so, AUX_SYS);
        __builtin_amdgcn_raw_buffer_store_b128(h[2 * t + 1], rsO, vo, so + 8 * NSH * 4, AUX_SYS);
      }
      STAMP(2 + ((j / NXP) & 7));
    }
    if (xi == NXP - 1) {
      const int vo = lane < NHQ ? lane * 16 : OOB;
      for (;;) {
        const v4u v = __builtin_amdgcn_raw_buffer_load_b128(rsS, vo, YF_OFF + w * YF_SRC, AUX_SYS);
        if (__all(lane >= NHQ || (v.x >= seq && v.y >= seq && v.z >= seq && v.w >= seq))) break;
        if (wt.give_up()) break;
        __builtin_amdgcn_s_sleep(2);
      }
    }
    if (wt.dead) __builtin_amdgcn_raw_buffer_store_b32(0x7fc00000u, rsO, vone, rank * (MSH * NSH * 4), AUX_SYS);
    STAMP(1);
    STAMPV(41, (int)qd);
    STAMP_OUT();
    return;
  }

  const bool helper = (qd & 4u) != 0u;
  const int hpan = (int)(qd & 3u), grp = (int)((qd >> 8) & 255u), ngrp = (int)((qd >> 16) & 255u), hq = (int)(qd >> 24);
  unsigned lst;
  int nun;
  {
    const int li = grp + lane * ngrp;
    const bool inr = li < (helper ? NHLP_U : NOWN_U);
    const unsigned e = a.list[helper ? 1 : 0][inr ? li : 0];
    const bool keep = inr && !(helper && (int)(e >> 5) == obase + hsrc);
    const unsigned long long km = __ballot(keep);
    nun = __builtin_popcountll(km);
    const int pos = keep ? (int)__builtin_amdgcn_mbcnt_hi((unsigned)(km >> 32), __builtin_amdgcn_mbcnt_lo((unsigned)km, 0u)) : 63;
    lst = (unsigned)__builtin_amdgcn_ds_permute(pos * 4, (int)e);
  }
  const unsigned dest = (unsigned)(helper ? obase + hsrc : rank);
  const int col0 = helper ? myidx * 128 + hpan * 32 : (hpan * 4 + hsrc) * 32;
  const char* const p_ws = a.ws[rank];
  const float* const p_out = a.out[dest & 7u];

  v4u b[64];
  v4u av[32];
  int eb[4];
  TAKE_SEQ();
  PUT_SEQ();
  __builtin_amdgcn_s_sleep(CFG_DELAY);
  {
    const int id = (hq * 4 + hsrc) * 4 + w;
    if (id < 256) {
      v4u h[16];
      const rsrc_t rs = mk_rsrc(helper ? a.x : a.w, X_BYTES);
      load_block(av, rs, helper ? x_vo(lane) : w_vo(lane), helper ? +X_VD : +W_VD, helper ? x_so(id >> 3, id & 7) : w_so(id & 15, (id >> 4) * 32),
                 helper ? +X_S0 : +W_S0, helper ? +X_S1 : +W_S1);
      int se;
      if (helper) se = cvt_x(av, h, wlds, lane);
      else se = cvt_w(av, h);
      const int doff = helper ? XA_OFF + par * XA_PAR + rank * 32 * UNIT_B + id * ITEM_B : WB_OFF + par * WB_PAR + (256 + id) * ITEM_B;
      const int foff = helper ? XF_OFF + par * XF_PAR + (rank * 32 + (id >> 3)) * 128 + (id & 7) * 4
                              : WF_OFF + par * WF_PAR + (16 + (id >> 4)) * 128 + (id & 15) * 4;
#pragma unroll
      for (int s = 0; s < 16; ++s)
        __builtin_amdgcn_raw_buffer_store_b128(h[s], rsS, lane * 16 + (s & 3) * 1024, doff + (s >> 2) * 4096, AUX_SYS);
      WAIT_VM(0);
      __builtin_amdgcn_raw_buffer_store_b32(seq * 256u + (unsigned)se, rsS, vone, foff, AUX_SYS);
    }
    STAMP(1);
    const int pidx = helper ? hsrc * 4 + hpan : 16 + hpan * 4 + hsrc;
    const int fo = WF_OFF + par * WF_PAR + pidx * 128 + w * 16;
    unsigned f0, f1, f2, f3;
    for (;;) {
      const v4u v = __builtin_amdgcn_raw_buffer_load_b128(rsS, 0, fo, AUX_SYS);
      f0 = rfl(v.x); f1 = rfl(v.y); f2 = rfl(v.z); f3 = rfl(v.w);
      if ((f0 >> 8) == seq && (f1 >> 8) == seq && (f2 >> 8) == seq && (f3 >> 8) == seq) break;
      if (wt.give_up()) break;
      __builtin_amdgcn_s_sleep(CFG_WSLEEP);
    }
    eb[0] = 254 - (int)(f0 & 255u); eb[1] = 254 - (int)(f1 & 255u); eb[2] = 254 - (int)(f2 & 255u); eb[3] = 254 - (int)(f3 & 255u);
    STAMP(2);
    const int bo = WB_OFF + par * WB_PAR + (pidx * 16 + w * 4) * ITEM_B;
#pragma unroll
    for (int s = 0; s < 64; ++s)
      b[s] = __builtin_amdgcn_raw_buffer_load_b128(rsS, lane * 16 + (s & 3) * 1024, bo + (s >> 2) * 4096, AUX_SYS);
    SCHED_FENCE();
  }
  __syncthreads();
  STAMP(3);

#pragma unroll
  for (int s = 0; s < CFG_NAGPR; ++s) asm volatile("" : "+a"(b[s]));

  const int l_a = XA_OFF + par * XA_PAR + w * (2 * ITEM_B) + (int)lst * UNIT_B;
  const int l_y = (int)lst * (16 * NSH * 4) + col0 * 4;
  const int l_f = XF_OFF + par * XF_PAR + w * 8 + (int)lst * 128;
  const unsigned l_d = seq << 12 | lst << 2 | (unsigned)hpan;
  const unsigned stmask = (unsigned)__ballot(
      helper && !((int)(lst >> 7) == (rank >> 2) && (int)(lst >> 5) != rank && (int)(lst & 31u) >= KU - NDIR));
  int vk[8];
#pragma unroll
  for (int k = 0; k < 8; ++k) vk[k] = lane * 16 + k * 4096;
  auto pin_vk = [&]() INL {
    asm volatile("" : "+v"(vk[0]), "+v"(vk[1]), "+v"(vk[2]), "+v"(vk[3]), "+v"(vk[4]), "+v"(vk[5]), "+v"(vk[6]), "+v"(vk[7]));
  };
  auto load_a = [&](int s, int ao) INL { return __builtin_amdgcn_raw_buffer_load_b128(rsS, vk[s >> 2] + (s & 3) * 1024, ao, AUX_SYS); };

  unsigned todo = (1u << nun) - 1u, watch = todo;
  unsigned rdy = 0, part = 0, mine = 0;
  unsigned exps = 0;
  float bad = wt.dead ? __builtin_nanf("") : 0.f;
  if (wt.dead) mine = 0x7fffffffu;
  const unsigned seq8 = seq << 8;
  v2u pv;
  auto poll_issue = [&]() INL {
    const int lf = l_f;
    pv = __builtin_amdgcn_raw_buffer_load_b64(rsS, __builtin_amdgcn_inverse_ballot_w64(watch & ~mine) ? lf : +OOB, 0, AUX_SYS);
  };
  auto poll_eval = [&]() INL {
    const bool ok = ((pv.x ^ seq8) | (pv.y ^ seq8)) < 256u;
    exps = ok ? __builtin_amdgcn_perm(pv.y, pv.x, 0x0c0c0400u) : exps;
    mine |= (unsigned)__ballot(ok);
  };

  int ld_w = w * RED_WAVE + (lane & 15) * 144 + ((lane >> 4) & 1) * 64 + (lane >> 5) * 8;
  int ld_r = (w * 4 + ((lane >> 3) & 3)) * 144 + (lane & 7) * 16;
  int ld_c = RED_BUF;
  const int vy = lane < 32 ? (w * 4 + (lane >> 3)) * (NSH * 4) + (lane & 7) * 16 : OOB;
  const int vs = lane < 32 ? w * 512 + lane * 16 : OOB;
  const int ylink = (hsrc - myidx) & 3;
  const unsigned ybase = (seq - 1u) * (unsigned)NST;
  const int ys0 = YS_OFF + ylink * YS_LINK, yd0 = YD_OFF + ylink * YD_LINK + w * 4;
  int vzero = 0;
  asm volatile("" : "+v"(vzero));
  int pend_vo = OOB;
  unsigned pend_val = 0;
  auto emit = [&]() INL {
    __builtin_amdgcn_raw_buffer_store_b32(pend_val, rsS, pend_vo, yd0, AUX_SYS);
    pend_vo = OOB;
  };
  auto post = [&](unsigned slot) INL { *(LAS v2u*)((LAS char*)lds + ld_c + w * 8) = v2u{mine, slot}; };
  auto share = [&](v4u m0, v4u m1) INL {
    rdy = rfl(m0.x & m0.z & m1.x & m1.z);
    part = rfl(m0.x | m0.z | m1.x | m1.z);
    ld_w ^= 16384;
    ld_r ^= 16384;
    ld_c ^= 16384;
  };
  auto choose = [&]() INL {
    const unsigned c2 = part & todo;
    unsigned c = rdy & todo;
    c = c ? c : c2;
    c = __builtin_popcount(todo) == 1 ? todo : c;
    const int i = __builtin_ctz(c | 0x80000000u);
    todo &= ~(1u << i);
    return i;
  };
  auto have = [&](int i) INL { return (mine >> i & 1u) != 0u; };
  auto repoll = [&]() INL {
    if (wt.give_up()) {
      mine = 0x7fffffffu;
      bad = __builtin_nanf("");
    }
    __builtin_amdgcn_s_sleep(CFG_PSLEEP);
    poll_issue();
    WAIT_VM(1);
    emit();
  };

  constexpr int NLOOP = CFG_NLOOP, NPRE = CFG_NPRE;
  int cnt = 0;
  int o_st = 0, o_so = 0;
  unsigned o_d = 0;
  auto front = [&](auto pf_tag, int icur, int inext) INL {
    constexpr bool PF = decltype(pf_tag)::value;
    CYC(52);
    pin_vk();
    const int prev_st = o_st;
    int ao = 0, tslot = 0, ex[4];
    f32x4 acc0 = {0.f, 0.f, 0.f, 0.f}, acc1 = {0.f, 0.f, 0.f, 0.f};
    f32x2 p[4];
#pragma unroll
    for (int j = 0; j < 4; ++j) p[j] = f32x2{bad, bad};
#pragma unroll
    for (int s = 0; s < 32; ++s) {
      if ((s & 7) == 0) CYC(53 + (s >> 3));
      if (s > 0 && (s & 7) == 0 && ex[(s >> 3) - 1] != ex[s >> 3]) {
#pragma unroll
        for (int j = 0; j < 4; ++j) {
          p[j][0] += __builtin_amdgcn_ldexpf(acc0[j], ex[(s >> 3) - 1]);
          p[j][1] += __builtin_amdgcn_ldexpf(acc1[j], ex[(s >> 3) - 1]);
        }
        acc0 = acc1 = f32x4{0.f, 0.f, 0.f, 0.f};
      }
      const v4u cur = av[s];
      acc0 = mfma(b[2 * s], cur, acc0);
      acc1 = mfma(b[2 * s + 1], cur, acc1);
      if (s == 0 && PF) {
        watch &= ~(1u << inext);
        ao = __builtin_amdgcn_readlane(l_a, inext);
        poll_issue();
      }
      if (s == 1) {
        o_st = (int)(stmask >> icur & 1u);
        if (o_st && threadIdx.x == 0) tslot = __builtin_amdgcn_raw_ptr_buffer_atomic_add_i32(1, rsS, vzero, YQ_OFF + ylink * 128, 0);
      }
      if (s == 3) {
        const unsigned ex2 = (unsigned)__builtin_amdgcn_readlane((int)exps, icur);
        const int ea0 = (int)(ex2 & 255u), ea1 = (int)(ex2 >> 8);
        ex[0] = eb[0] - ea0; ex[1] = eb[1] - ea0; ex[2] = eb[2] - ea1; ex[3] = eb[3] - ea1;
        asm volatile("" : "+s"(ex[0]), "+s"(ex[1]), "+s"(ex[2]), "+s"(ex[3]));
      }
      if (s == 5) {
        o_so = __builtin_amdgcn_readlane(l_y, icur);
        o_d = (unsigned)__builtin_amdgcn_readlane((int)l_d, icur);
        asm volatile("" : "+s"(o_so), "+s"(o_d));
      }
      if (PF && s >= 2 && (s - 1) * NLOOP / 30 > (s - 2) * NLOOP / 30) av[(s - 2) * NLOOP / 30] = load_a((s - 2) * NLOOP / 30, ao);
      if (!PF && s == CFG_POLL_AT) poll_issue();
      if (PF && s == 29) {
        WAIT_VM(28 * NLOOP / 30);
        poll_eval();
        emit();
        post(rfl((unsigned)tslot) - ybase);
      }
      SCHED_FENCE();
    }
    if (!PF) STAMP(47);
    CYC(57);
#pragma unroll
    for (int j = 0; j < 4; ++j) {
      if (PF && j < NPRE) av[NLOOP + j] = load_a(NLOOP + j, ao);
      SCHED_FENCE();
      p[j][0] += __builtin_amdgcn_ldexpf(acc0[j], ex[3]);
      p[j][1] += __builtin_amdgcn_ldexpf(acc1[j], ex[3]);
      *(LAS f32x2*)((LAS char*)lds + ld_w + j * 16) = p[j];
      SCHED_FENCE();
    }
    CYC(58);
    if (!PF && (watch != 0u || o_st || prev_st)) {
      WAIT_VM(0);
      poll_eval();
      emit();
      post(rfl((unsigned)tslot) - ybase);
    }
    SCHED_FENCE();
    return ao;
  };
  auto back = [&](auto pf_tag, int ao) INL {
    constexpr bool PF = decltype(pf_tag)::value;
    constexpr int NB = 32 - NLOOP - NPRE;
    auto more = [&](auto k0, auto k1) INL {
      SCHED_FENCE();
      if (PF) {
#pragma unroll
        for (int s = NLOOP + NPRE + NB * decltype(k0)::value / 4; s < NLOOP + NPRE + NB * decltype(k1)::value / 4; ++s) av[s] = load_a(s, ao);
      }
      SCHED_FENCE();
    };
#define MORE(k0, k1) more(std::integral_constant<int, k0>(), std::integral_constant<int, k1>())
    CYC(59);
    WAIT_LGKM0();
    __builtin_amdgcn_s_barrier();
    SCHED_FENCE();
    CYC(60);
    const LAS char* const rb = (const LAS char*)lds + ld_r;
    const v4u m0 = *(const LAS v4u*)((LAS char*)lds + ld_c), m1 = *(const LAS v4u*)((LAS char*)lds + ld_c + 16);
    const f32x4 p0 = *(const LAS f32x4*)rb, p1 = *(const LAS f32x4*)(rb + RED_WAVE);
    const f32x4 p2 = *(const LAS f32x4*)(rb + 2 * RED_WAVE), p3 = *(const LAS f32x4*)(rb + 3 * RED_WAVE);
    MORE(0, 1);
    const int st = (int)rfl((unsigned)o_st), vs0 = vs, vy0 = vy, vone0 = vone;
    const rsrc_t rs = mk_rsrc(st ? (const void*)p_ws : (const void*)p_out, st ? WS_USED : OUT_BYTES);
    int vo = st ? vs0 : vy0;
    pend_vo = st ? vone0 : +OOB;
    pend_val = o_d;
    asm volatile("" : "+v"(vo), "+v"(pend_vo), "+v"(pend_val));
    MORE(1, 2);
    share(m0, m1);
    const int ts = (int)(rfl(m0.y) & 511u);
    int so = (int)rfl((unsigned)(st ? ys0 + ts * 2048 : +o_so));
    pend_vo = (int)((unsigned)pend_vo + (unsigned)ts * 16u);
    asm volatile("" : "+s"(rdy), "+s"(part), "+s"(so));
    MORE(2, 3);
    f32x4 y = (p0 + p1) + (p2 + p3);
    y.x = y.x <= 0.f ? 0.f : y.x;
    y.y = y.y <= 0.f ? 0.f : y.y;
    y.z = y.z <= 0.f ? 0.f : y.z;
    y.w = y.w <= 0.f ? 0.f : y.w;
    __builtin_amdgcn_raw_buffer_store_b128(__builtin_bit_cast(v4u, y), rs, vo, so, AUX_SYS);
    MORE(3, 4);
    CYC(61);
  };

  int icur, ci = 31;
  bool defer = false;
  poll_issue();
  emit();
  for (;;) {
    while (ci == 31) {
      poll_eval();
      post(0u);
      WAIT_LGKM0();
      __builtin_amdgcn_s_barrier();
      share(*(const LAS v4u*)((LAS char*)lds + ld_c), *(const LAS v4u*)((LAS char*)lds + ld_c + 16));
      ci = choose();
      if (!have(ci)) repoll();
    }
    while (!have(ci)) {
      poll_eval();
      if (!have(ci)) repoll();
    }
    icur = ci;
    watch &= ~(1u << icur);
    STAMP(44);
    STAMPV(46, __builtin_amdgcn_readlane((int)lst, icur));
    {
      const int ao = __builtin_amdgcn_readlane(l_a, icur);
      pin_vk();
#pragma unroll
      for (int s = 0; s < 32; ++s) {
        av[s] = load_a(s, ao);
        SCHED_FENCE();
      }
    }
    STAMP(45);
    if (defer) {
      back(std::false_type(), 0);
    } else {
      WAIT_VM(32);
      emit();
    }
    ci = choose();
    while (have(ci)) {
      const int ao = front(std::true_type(), icur, ci);
      back(std::true_type(), ao);
      STAMP(4 + cnt);
      ++cnt;
      icur = ci;
      ci = choose();
    }
    front(std::false_type(), icur, 0);
    STAMP(4 + cnt);
    ++cnt;
    defer = have(ci);
    if (!defer) {
      if (watch != 0u) poll_issue();
      back(std::false_type(), 0);
      if (watch == 0u) break;
      if (ci == 31) ci = choose();
    }
  }
  (void)cnt;
  WAIT_VM(0);
  emit();
  if (helper) {
    const rsrc_t rsD = mk_rsrc(a.ws[dest & 7u], WS_USED);
    __builtin_amdgcn_raw_buffer_store_b32(seq, rsD, vone, YF_OFF + myidx * YF_SRC + (hq * 4 + w) * 4, AUX_SYS);
  }
  STAMP(40);
  STAMPV(51, (int)(unsigned)__builtin_amdgcn_s_memtime());
  STAMPV(41, (int)qd);
  STAMP_OUT();
}

struct Ev { double t; int e; };

}

extern "C" void kernel_launch(int rank, int nranks, ncclComm_t comm,
                              hipStream_t stream,
                              void* const* d_in, const int* in_sizes, int n_in,
                              void* d_out, int out_size,
                              const int* in_dtypes, int out_dtype,
                              void* const* d_out_all,
                              void* const* d_ws_all, size_t ws_bytes) {
  (void)comm; (void)in_dtypes; (void)out_dtype; (void)d_out;
  if (nranks != NRANK || n_in < 2 || (unsigned)rank >= (unsigned)NRANK || ws_bytes < (size_t)WS_USED ||
      in_sizes[0] != MSH * KDIM || in_sizes[1] != KDIM * NSH || out_size != NRANK * MSH * NSH) {
    fprintf(stderr, "unexpected problem shape\n");
    return;
  }
  Args a = {};
  a.x = (const float*)d_in[0];
  a.w = (const float*)d_in[1];
  for (int p = 0; p < NRANK; ++p) {
    a.ws[p] = (char*)d_ws_all[p];
    a.out[p] = (float*)d_out_all[p];
  }
  a.rank = rank;

  const int gbase = rank & 4, obase = gbase ^ 4;
  std::vector<Ev> ev;
  for (int u = 0; u < 32; ++u) ev.push_back({0.001 * u, rank * 32 + u});
  for (int p = gbase; p < gbase + 4; ++p)
    if (p != rank)
      for (int u = 0; u < KU; ++u) ev.push_back({1.0 + u + 0.01 * p, p * 32 + u});
  for (int p = obase; p < obase + 4; ++p)
    for (int v = 0; v < GU; ++v) ev.push_back({9.1 + v + 0.01 * p, p * 32 + KU + v});
  std::stable_sort(ev.begin(), ev.end(), [](const Ev& l, const Ev& r) { return l.t < r.t; });
  int n0 = 0, n1 = 0;
  for (const Ev& v : ev) {
    a.list[0][n0++] = (unsigned char)v.e;
    if (!((v.e >> 5) == rank && (v.e & 31) >= KU)) a.list[1][n1++] = (unsigned char)v.e;
  }
  const int gh[4] = CFG_GH;
  static_assert(NHQ <= 32 && NHQ >= 16 && GOWN >= 4, "done flags of one helper fit 512 bytes; 256 blocks of x and of w are converted");
  if (gh[0] + gh[1] + gh[2] + gh[3] != NHQ || n0 != NOWN_U || n1 != NHLP_U) { fprintf(stderr, "bad quad split\n"); return; }
  int used[8] = {0, 0, 0, 0, 0, 0, 0, 0};
  for (int c = 0; c < 4; ++c)
    for (int g = 0; g < GOWN; ++g) {
      const int x = c + 4 * (g & 1);
      a.quad[x * 6 + used[x]++] = (unsigned)c | (unsigned)g << 8 | (unsigned)GOWN << 16 | (unsigned)(g * 4 + c) << 24;
    }
  for (int q = 0; q < NXQ; ++q) a.quad[(7 - q) * 6 + used[7 - q]++] = 8u | (unsigned)q << 24;
  int hq = 0, x = 0;
  for (int p = 3; p >= 0; --p)
    for (int g = 0; g < gh[p]; ++g) {
      if ((NHLP_U + gh[p] - 1) / gh[p] > LMAX) { fprintf(stderr, "bad quad split\n"); return; }
      while (used[x] >= 6) x = (x + 1) & 7;
      a.quad[x * 6 + used[x]++] = (unsigned)(4 + p) | (unsigned)g << 8 | (unsigned)gh[p] << 16 | (unsigned)hq << 24;
      ++hq;
      x = (x + 1) & 7;
    }
  static_assert((NOWN_U + GOWN - 1) / GOWN <= LMAX, "one list entry per bit");
  hipLaunchKernelGGL(ag_gemm_kernel, dim3(256), dim3(256), 0, stream, a);
  HIP_CHECK(hipGetLastError());
}
```

```cpp
#include <hip/hip_runtime.h>
#include <rccl/rccl.h>
#include <algorithm>
#include <cstdio>
#include <type_traits>
#include <vector>

#define HIP_CHECK(expr) do { hipError_t _e = (expr); if (_e != hipSuccess) { \
    fprintf(stderr, "HIP error %s at %s:%d\n", hipGetErrorString(_e), __FILE__, __LINE__); \
    return; } } while (0)

namespace {

constexpr int NRANK = 8;
constexpr int KDIM = 4096, MSH = 512, NSH = 512;
constexpr int KU = 22, GU = 32 - KU;
constexpr int GOWN = 5, NQUAD = 48, NXQ = 2, NHQ = NQUAD - 4 * GOWN - NXQ;
constexpr int NXP = 8 + 4 * NXQ;
constexpr int NOWN_U = 32 + 3 * KU + 4 * GU, NHLP_U = 4 * KU + 4 * GU;
constexpr int UNIT_B = 16 * KDIM * 2, ITEM_B = 16384;

constexpr int DEPTH = 2, DEPTH0 = 5, DEPTHC = 3, TOKEN_AT = 7, CLAIM_AT = 8;
constexpr int DS0 = 7, DS0Y = 4, DS1 = 4, LPACE = 4, STAGGER = 12;
constexpr int NRAW = 6;
constexpr int DELAY = 48, NLOOP = 20, NPRE = 4, POLL_AT = 12, NAGPR = 16;

constexpr int NDIR = 4, NST = 4 * (NHLP_U - GU - 3 * NDIR), NIT_Y = (NST + 7) / 8;
constexpr int NIT_IN = 8 * KU, NIT_X = 64 + 8 * GU, NIT_C = NIT_X + NIT_Y;
static_assert(NIT_Y <= 64 && NRAW >= 1, "the last, possibly short block travels unpacked");

constexpr int XA_OFF = 0, XA_PAR = NRANK * 32 * UNIT_B;
constexpr int WB_OFF = XA_OFF + 2 * XA_PAR, WB_PAR = 32 * 16 * ITEM_B;
constexpr int YS_OFF = WB_OFF + 2 * WB_PAR, YS_LINK = 64 * ITEM_B;
constexpr int YC_OFF = YS_OFF + 4 * YS_LINK;
constexpr int YD_OFF = YC_OFF + 4 * YS_LINK, YD_LINK = 64 * 128;
constexpr int XF_OFF = YD_OFF + 4 * YD_LINK, XF_PAR = NRANK * 32 * 128;
constexpr int WF_OFF = XF_OFF + 2 * XF_PAR, WF_PAR = 32 * 128;
constexpr int YF_OFF = WF_OFF + 2 * WF_PAR;
constexpr int YF_SRC = 1024;
constexpr int CNT_OFF = YF_OFF + 4 * YF_SRC;
constexpr int TMO_OFF = CNT_OFF + 1024 * 4;
constexpr int QC_OFF = TMO_OFF + 1024, YQ_OFF = QC_OFF + 1024;
constexpr int WS_USED = TMO_OFF + 4096;

constexpr int X_BYTES = MSH * KDIM * 4, W_BYTES = KDIM * NSH * 4, OUT_BYTES = NRANK * MSH * NSH * 4;
static_assert(X_BYTES == W_BYTES, "one descriptor size serves both inputs");
constexpr int RSRC_W3 = 0x00020000;
constexpr int AUX_SYS = 17;
constexpr int OOB = 0x7ffffff0;
constexpr unsigned long long DEADLINE_TICKS = 150000000ull;

typedef unsigned v4u __attribute__((ext_vector_type(4)));
typedef unsigned v2u __attribute__((ext_vector_type(2)));
typedef float f32x4 __attribute__((ext_vector_type(4)));
typedef float f32x2 __attribute__((ext_vector_type(2)));
typedef _Float16 h8 __attribute__((ext_vector_type(8)));
typedef _Float16 h2 __attribute__((ext_vector_type(2)));
typedef __amdgpu_buffer_rsrc_t rsrc_t;

struct Args {
  const float* x;
  const float* w;
  char* ws[NRANK];
  float* out[NRANK];
  int rank;
  unsigned quad[NQUAD];
  unsigned char list[2][160];
};

#define INL __attribute__((always_inline))
#define LAS __attribute__((address_space(3)))
#define SCHED_FENCE() __builtin_amdgcn_sched_barrier(0)
#define WAIT_VM(n) asm volatile("s_waitcnt vmcnt(%0)" ::"n"(n) : "memory")
#define WAIT_LGKM0() asm volatile("s_waitcnt lgkmcnt(0)" ::: "memory")
#define LDS_BARRIER() do { WAIT_LGKM0(); __builtin_amdgcn_s_barrier(); asm volatile("" ::: "memory"); } while (0)

__device__ __forceinline__ rsrc_t mk_rsrc(const void* p, int bytes) {
  return __builtin_amdgcn_make_buffer_rsrc((void*)p, 0, bytes, RSRC_W3);
}
__device__ __forceinline__ unsigned rfl(unsigned v) { return (unsigned)__builtin_amdgcn_readfirstlane((int)v); }
__device__ __forceinline__ unsigned pk_f16(float a, float b) {
  h2 v = {(_Float16)a, (_Float16)b};
  return __builtin_bit_cast(unsigned, v);
}
__device__ __forceinline__ f32x4 mfma(v4u a, v4u b, f32x4 c) {
  return __builtin_amdgcn_mfma_f32_16x16x32_f16(__builtin_bit_cast(h8, a), __builtin_bit_cast(h8, b), c, 0, 0, 0);
}
template <int CTRL, int ROWMASK>
__device__ __forceinline__ unsigned dpp_max(unsigned m) {
  const unsigned o = (unsigned)__builtin_amdgcn_update_dpp((int)m, (int)m, CTRL, ROWMASK, 0xf, false);
  return o > m ? o : m;
}

__device__ __forceinline__ int block_scale(const v4u (&f)[32]) {
  float m = 0.f;
#pragma unroll
  for (int i = 0; i < 32; ++i) {
    asm volatile("v_max3_f32 %0, |%1|, |%2|, %0\n\tv_max3_f32 %0, |%3|, |%4|, %0"
                 : "+v"(m) : "v"(f[i].x), "v"(f[i].y), "v"(f[i].z), "v"(f[i].w));
  }
  asm volatile("s_nop 1" : "+v"(m));
  unsigned u = __builtin_bit_cast(unsigned, m) & 0x7fffffffu;
  u = dpp_max<0xB1, 0xf>(u);
  u = dpp_max<0x4E, 0xf>(u);
  u = dpp_max<0x141, 0xf>(u);
  u = dpp_max<0x128, 0xf>(u);
  const unsigned m0 = (unsigned)__builtin_amdgcn_readlane((int)u, 0), m1 = (unsigned)__builtin_amdgcn_readlane((int)u, 16);
  const unsigned m2 = (unsigned)__builtin_amdgcn_readlane((int)u, 32), m3 = (unsigned)__builtin_amdgcn_readlane((int)u, 48);
  const unsigned mm = max(max(m0, m1), max(m2, m3));
  int se = 127 + 14 + 127 - (int)((mm >> 23) | 7u);
  return se > 254 ? 254 : se;
}

template <int PACE = 0>
__device__ __forceinline__ void load_block(v4u (&f)[32], rsrc_t rs, int vo, int vd, int so, int s0, int s1) {
#pragma unroll
  for (int m = 0; m < 16; ++m) {
    f[2 * m] = __builtin_amdgcn_raw_buffer_load_b128(rs, vo, so + (m >> 2) * s1 + (m & 3) * s0, 0);
    f[2 * m + 1] = __builtin_amdgcn_raw_buffer_load_b128(rs, vo + vd, so + (m >> 2) * s1 + (m & 3) * s0, 0);
    SCHED_FENCE();
    if (PACE > 0) {
      __builtin_amdgcn_s_sleep(PACE);
      SCHED_FENCE();
    }
  }
  SCHED_FENCE();
}
constexpr int X_VD = 16, X_S0 = KDIM * 4, X_S1 = 4 * KDIM * 4;
__device__ __forceinline__ int x_vo(int lane) { return lane * 32; }
__device__ __forceinline__ int x_so(int unit, int seg) { return unit * (16 * KDIM * 4) + seg * 2048; }
constexpr int W_VD = NSH * 4, W_S0 = 2 * NSH * 4, W_S1 = 64 * NSH * 4;
__device__ __forceinline__ int w_vo(int lane) { return (((lane >> 3) & 1) * 32 + (lane >> 4) * 8) * (NSH * 4) + (lane & 7) * 16; }
__device__ __forceinline__ int w_so(int chunk, int col0) { return chunk * (256 * NSH * 4) + col0 * 4; }

__device__ __forceinline__ int cvt_x(const v4u (&f)[32], v4u (&h)[16], char* lds, int lane) {
  const int se = block_scale(f);
  const float sc = __builtin_bit_cast(float, (unsigned)se << 23);
#pragma unroll
  for (int r = 0; r < 16; ++r) {
    const f32x4 a = __builtin_bit_cast(f32x4, f[2 * r]) * sc, b = __builtin_bit_cast(f32x4, f[2 * r + 1]) * sc;
    const v4u p = {pk_f16(a.x, a.y), pk_f16(a.z, a.w), pk_f16(b.x, b.y), pk_f16(b.z, b.w)};
    *(v4u*)(lds + lane * 256 + ((r ^ (lane & 15)) << 4)) = p;
  }
  const int q = lane >> 4, r = lane & 15;
#pragma unroll
  for (int s = 0; s < 16; ++s) {
    const int l = 4 * s + q;
    h[s] = *(const v4u*)(lds + l * 256 + ((r ^ (l & 15)) << 4));
  }
  return se;
}

__device__ __forceinline__ int cvt_w(const v4u (&f)[32], v4u (&h)[16]) {
  const int se = block_scale(f);
  const float sc = __builtin_bit_cast(float, (unsigned)se << 23);
#pragma unroll
  for (int t = 0; t < 4; ++t) {
    v4u p[4];
#pragma unroll
    for (int jj = 0; jj < 4; ++jj) {
      const f32x4 a = __builtin_bit_cast(f32x4, f[8 * t + 2 * jj]) * sc, b = __builtin_bit_cast(f32x4, f[8 * t + 2 * jj + 1]) * sc;
      p[0][jj] = pk_f16(a.x, b.x);
      p[1][jj] = pk_f16(a.y, b.y);
      p[2][jj] = pk_f16(a.z, b.z);
      p[3][jj] = pk_f16(a.w, b.w);
    }
#pragma unroll
    for (int c = 0; c < 2; ++c)
#pragma unroll
      for (int jj = 0; jj < 4; ++jj) {
        h[4 * t + c][jj] = (unsigned)__builtin_amdgcn_update_dpp((int)p[c][jj], (int)p[c + 2][jj], 0x128, 0xf, 0xc, false);
        h[4 * t + 2 + c][jj] = (unsigned)__builtin_amdgcn_update_dpp((int)p[c + 2][jj], (int)p[c][jj], 0x128, 0xf, 0x3, false);
      }
  }
  return se;
}

constexpr int PK_DESC = 512, PK_DATA = 544, PK_MAX = 14;
template <int CTRL, int ROWMASK>
__device__ __forceinline__ int dpp_add(int v) { return v + __builtin_amdgcn_update_dpp(0, v, CTRL, ROWMASK, 0xf, false); }
__device__ __forceinline__ int wave_scan(int v) {
  v = dpp_add<0x111, 0xf>(v);
  v = dpp_add<0x112, 0xf>(v);
  v = dpp_add<0x114, 0xf>(v);
  v = dpp_add<0x118, 0xf>(v);
  v = dpp_add<0x142, 0xa>(v);
  v = dpp_add<0x143, 0xc>(v);
  return v;
}

struct Waiter {
  rsrc_t rs;
  unsigned seq;
  unsigned long long t0;
  unsigned spins;
  bool dead;
  __device__ __forceinline__ bool give_up() {
    if (dead) return true;
    if ((++spins & 63u) != 0u) return false;
    const unsigned t = rfl(__builtin_amdgcn_raw_buffer_load_b32(rs, 0, TMO_OFF, AUX_SYS));
    if (t == seq || __builtin_amdgcn_s_memrealtime() - t0 > DEADLINE_TICKS) {
      __builtin_amdgcn_raw_buffer_store_b32(seq, rs, 0, TMO_OFF, AUX_SYS);
      dead = true;
      return true;
    }
    return false;
  }
};

__device__ __forceinline__ unsigned fetch_block(v4u (&h)[16], Waiter& wt, int link, int j, int lane) {
  const bool need = lane < 32 && 8 * j + (lane >> 2) < NST;
  unsigned d;
  for (;;) {
    d = __builtin_amdgcn_raw_buffer_load_b32(wt.rs, need ? lane * 4 : +OOB, YD_OFF + link * YD_LINK + j * 128, AUX_SYS);
    if (__all(!need || (d >> 12) == (wt.seq & 0xfffffu))) break;
    if (wt.give_up()) break;
    __builtin_amdgcn_s_sleep(1);
  }
#pragma unroll
  for (int s = 0; s < 16; ++s)
    h[s] = __builtin_amdgcn_raw_buffer_load_b128(wt.rs, lane * 16 + (s & 3) * 1024, YS_OFF + link * YS_LINK + j * ITEM_B + (s >> 2) * 4096, AUX_SYS);
  return d;
}
__device__ __forceinline__ int pack_block(const v4u (&h)[16], unsigned d, LAS char* pk, int lane) {
  unsigned mk[2] = {0u, 0u};
#pragma unroll
  for (int r = 0; r < 64; ++r) mk[r >> 5] |= (h[r >> 2][r & 3] != 0u ? 1u : 0u) << (r & 31);
  const int cnt = __builtin_popcount(mk[0]) + __builtin_popcount(mk[1]) + 1;
  const int inc = wave_scan(cnt);
  const int total = PK_DATA + 4 * __builtin_amdgcn_readlane(inc, 63);
  if (total > PK_MAX * 1024) return 0;
  *(LAS v2u*)(pk + lane * 8) = v2u{mk[0], mk[1]};
  if ((lane & 3) == 0 && lane < 32) *(LAS unsigned*)(pk + PK_DESC + lane) = d;
  int ptr = PK_DATA + 4 * (inc - cnt);
#pragma unroll
  for (int r = 0; r < 64; ++r) {
    *(LAS unsigned*)(pk + ptr) = h[r >> 2][r & 3];
    ptr += (int)(mk[r >> 5] >> (r & 31) & 1u) * 4;
  }
  return total;
}
__device__ __forceinline__ void load_packed(rsrc_t rs, int off, int np, LAS char* pk, int lane) {
  v4u t[PK_MAX];
#pragma unroll
  for (int p = 0; p < PK_MAX; ++p) t[p] = __builtin_amdgcn_raw_buffer_load_b128(rs, p < np ? lane * 16 : +OOB, off + p * 1024, AUX_SYS);
#pragma unroll
  for (int p = 0; p < PK_MAX; ++p) *(LAS v4u*)(pk + p * 1024 + lane * 16) = t[p];
}

constexpr int RED_WAVE = 16 * 144, RED_BUF = 4 * RED_WAVE;
constexpr int LMAX = 31;
constexpr int LDS_CTL = 4 * 16384;
constexpr int LDS_BYTES = LDS_CTL + 128;
static_assert(RED_BUF + 32 <= 16384, "reduction buffers reuse the transpose areas");

__global__ void __launch_bounds__(256, 1) ag_gemm_kernel(const Args a) {
  __shared__ __attribute__((aligned(16))) char lds[LDS_BYTES];
  const int lane = threadIdx.x & 63, w = (int)rfl(threadIdx.x >> 6);
  const int rank = a.rank, myidx = rank & 3, gbase = rank & 4, obase = gbase ^ 4;
  const int xcd = blockIdx.x & 7, slot = blockIdx.x >> 3;
  const rsrc_t rsS = mk_rsrc(a.ws[rank], WS_USED);
  const int vone = lane == 0 ? 0 : OOB;
  char* const wlds = lds + w * 16384;
  LAS int* const token = (LAS int*)((LAS char*)lds + LDS_CTL);

  const int cnt_off = CNT_OFF + (blockIdx.x * 4 + w) * 4;
  const unsigned cnt_v = __builtin_amdgcn_raw_buffer_load_b32(rsS, 0, cnt_off, AUX_SYS);
  Waiter wt;
  wt.rs = rsS;
  wt.t0 = __builtin_amdgcn_s_memrealtime();
  wt.spins = 0;
  wt.dead = false;
#define TAKE_SEQ() \
  const unsigned seq = rfl(cnt_v) + 1u; \
  const int par = (int)(seq & 1u); \
  wt.seq = seq
#define PUT_SEQ() __builtin_amdgcn_raw_buffer_store_b32(seq, rsS, vone, cnt_off, AUX_SYS)

  if (slot < 7) {
    const bool cross = slot >= 3;
    const unsigned peer = (unsigned)(cross ? obase + ((myidx + slot - 3) & 3) : gbase + ((myidx + 1 + slot) & 3));
    const int nit = cross ? NIT_C : NIT_IN, nld = cross ? NIT_X : NIT_IN;
    const rsrc_t rsD = mk_rsrc(a.ws[peer & 7u], WS_USED);
    v4u f[32], h[16];
    auto issue = [&](int n, auto pace) INL {
      const bool lw = cross && n < 64;
      const rsrc_t rs = mk_rsrc(lw ? a.w : a.x, X_BYTES);
      const int so = lw ? w_so(n & 15, (int)(peer & 3u) * 128 + (n >> 4) * 32) : x_so(((cross ? KU - 8 : 0) + (n >> 3)) & 31, n & 7);
      load_block<decltype(pace)::value>(f, rs, lw ? w_vo(lane) : x_vo(lane), lw ? W_VD : X_VD, so, lw ? W_S0 : X_S0, lw ? W_S1 : X_S1);
    };
    int vzero = 0;
    asm volatile("" : "+v"(vzero));
    int n = w * 8 + xcd;
    if (threadIdx.x == 0) *token = 0;
    LDS_BARRIER();
    for (int i = 0; i < w; ++i) __builtin_amdgcn_s_sleep(STAGGER);
    issue(n, std::integral_constant<int, 0>());
    TAKE_SEQ();
    const unsigned qbase = (seq - 1u) * (unsigned)nit - 32u;
    int turn = w;
    auto send = [&](rsrc_t rs, int vo, const int (&base)[8], int step, int foff, unsigned fval) INL {
      WAIT_LGKM0();
      while ((int)rfl(__hip_atomic_load(token, __ATOMIC_RELAXED, __HIP_MEMORY_SCOPE_WORKGROUP)) < turn) __builtin_amdgcn_s_sleep(1);
      SCHED_FENCE();
      int claim = 0;
      const int deep = (cross ? DS0Y : DS0) + ((nit - n) * DS1 >> 3);
#pragma unroll
      for (int s = 0; s < 16; ++s) {
        if (s == CLAIM_AT && lane == 0) claim = __builtin_amdgcn_raw_ptr_buffer_atomic_add_i32(1, rsS, vzero, QC_OFF + slot * 128, 0);
        __builtin_amdgcn_raw_buffer_store_b128(h[s], rs, vo, base[s >> 1] + (s & 1) * step, AUX_SYS);
        if (s == TOKEN_AT) __hip_atomic_fetch_max(token, turn + 1, __ATOMIC_RELAXED, __HIP_MEMORY_SCOPE_WORKGROUP);
        if (s < TOKEN_AT && turn == 0) WAIT_VM(DEPTH0);
        else if (s < 15 && s < deep) WAIT_VM(DEPTH);
        SCHED_FENCE();
      }
      WAIT_VM(0);
      __builtin_amdgcn_raw_buffer_store_b32(fval, rsD, vone, foff, AUX_SYS);
      SCHED_FENCE();
      const unsigned nn = rfl((unsigned)claim) - qbase;
      n = nn < (unsigned)nit ? (int)nn : nit;
      turn += 4;
    };
    for (;;) {
      const bool isw = cross && n < 64;
      const int unit = (cross ? KU - 8 : 0) + (n >> 3);
      int se;
      if (isw) se = cvt_w(f, h);
      else se = cvt_x(f, h, wlds, lane);
      const int doff = isw ? WB_OFF + par * WB_PAR + (myidx * 64 + n) * ITEM_B
                           : XA_OFF + par * XA_PAR + (rank * 32 + unit) * UNIT_B + (n & 7) * ITEM_B;
      const int foff = isw ? WF_OFF + par * WF_PAR + (myidx * 4 + (n >> 4)) * 128 + (n & 15) * 4
                           : XF_OFF + par * XF_PAR + (rank * 32 + unit) * 128 + (n & 7) * 4;
      const int base[8] = {doff, doff + 2048, doff + 4096, doff + 6144, doff + 8192, doff + 10240, doff + 12288, doff + 14336};
      send(rsD, lane * 16, base, 1024, foff, seq * 256u + (unsigned)se);
      if (n >= nld) break;
      issue(n, std::integral_constant<int, LPACE>());
    }
    const rsrc_t rsO = mk_rsrc(a.out[peer & 7u], OUT_BYTES);
    while (n < nit) {
      const int j = n - NIT_X;
      const int foff = YF_OFF + myidx * YF_SRC + 512 + j * 4;
      LAS char* const pk = (LAS char*)lds + w * 16384;
      const unsigned d = fetch_block(h, wt, slot - 3, j, lane);
      const int total = j < NIT_Y - NRAW && !wt.dead ? pack_block(h, d, pk, lane) : 0;
      if (total) {
        const int np = (total + 1023) >> 10, doff = YC_OFF + myidx * YS_LINK + j * ITEM_B;
        int claim = 0;
        if (lane == 0) claim = __builtin_amdgcn_raw_ptr_buffer_atomic_add_i32(1, rsS, vzero, QC_OFF + slot * 128, 0);
        while ((int)rfl(__hip_atomic_load(token, __ATOMIC_RELAXED, __HIP_MEMORY_SCOPE_WORKGROUP)) < turn) __builtin_amdgcn_s_sleep(1);
        for (int p = 0; p < np; ++p) {
          const v4u v = *(const LAS v4u*)(pk + p * 1024 + lane * 16);
          __builtin_amdgcn_raw_buffer_store_b128(v, rsD, p * 1024 + lane * 16 < total ? lane * 16 : +OOB, doff + p * 1024, AUX_SYS);
          if (p == (np - 1) / 2) __hip_atomic_fetch_max(token, turn + 1, __ATOMIC_RELAXED, __HIP_MEMORY_SCOPE_WORKGROUP);
          WAIT_VM(DEPTHC);
        }
        WAIT_VM(0);
        __builtin_amdgcn_raw_buffer_store_b32(seq << 5 | (unsigned)np, rsD, vone, foff, AUX_SYS);
        const unsigned nn = rfl((unsigned)claim) - qbase;
        n = nn < (unsigned)nit ? (int)nn : nit;
        turn += 4;
        continue;
      }
      int base[8];
#pragma unroll
      for (int t = 0; t < 8; ++t) {
        const unsigned dt = (unsigned)__builtin_amdgcn_readlane((int)d, 4 * t);
        base[t] = (8 * j + t < NST && !wt.dead) ? (int)((dt >> 2) & 255u) * (16 * NSH * 4) + (myidx * 128 + (int)(dt & 3u) * 32) * 4 : OUT_BYTES;
      }
      WAIT_VM(0);
      send(rsO, (lane >> 3) * (NSH * 4) + (lane & 7) * 16, base, 8 * NSH * 4, foff, wt.dead ? 0u : seq << 5);
    }
    __hip_atomic_fetch_max(token, 0x7fffffff, __ATOMIC_RELAXED, __HIP_MEMORY_SCOPE_WORKGROUP);
    PUT_SEQ();
    return;
  }

  const int cj = slot - 8, hsrc = cj & 3;
  const unsigned qd = slot == 7 ? 8u : a.quad[xcd * 6 + (cj >> 2)];
  if (qd & 8u) {
    TAKE_SEQ();
    (void)par;
    PUT_SEQ();
    const int xi = slot == 7 ? xcd : 8 + (int)(qd >> 24) * 4 + hsrc;
    const rsrc_t rsO = mk_rsrc(a.out[rank], OUT_BYTES);
    LAS char* const pk = (LAS char*)lds + w * 16384;
    for (int j = xi; j < NIT_Y; j += NXP) {
      unsigned fl;
      for (;;) {
        fl = rfl(__builtin_amdgcn_raw_buffer_load_b32(rsS, 0, YF_OFF + w * YF_SRC + 512 + j * 4, AUX_SYS));
        if ((fl >> 5) == (seq & 0x7ffffffu)) break;
        if (wt.give_up()) break;
        __builtin_amdgcn_s_sleep(2);
      }
      const int np = wt.dead ? 0 : (int)(fl & 31u);
      if (np == 0) continue;
      v4u h[16];
      load_packed(rsS, YC_OFF + w * YS_LINK + j * ITEM_B, np, pk, lane);
      const v2u mk = *(const LAS v2u*)(pk + lane * 8);
      const unsigned d = *(const LAS unsigned*)(pk + PK_DESC + (lane & 7) * 4);
      const int cnt = __builtin_popcount(mk.x) + __builtin_popcount(mk.y) + 1;
      int ptr = PK_DATA + 4 * (wave_scan(cnt) - cnt);
#pragma unroll
      for (int r = 0; r < 64; ++r) {
        const int m = (int)(mk[r >> 5] << (31 - (r & 31))) >> 31;
        h[r >> 2][r & 3] = *(const LAS unsigned*)(pk + (ptr & 16380)) & (unsigned)m;
        ptr -= 4 * m;
      }
#pragma unroll
      for (int t = 0; t < 8; ++t) {
        const unsigned dt = (unsigned)__builtin_amdgcn_readlane((int)d, t);
        const int so = (int)((dt >> 2) & 255u) * (16 * NSH * 4) + (w * 128 + (int)(dt & 3u) * 32) * 4;
        const int vo = (lane >> 3) * (NSH * 4) + (lane & 7) * 16;
        __builtin_amdgcn_raw_buffer_store_b128(h[2 * t], rsO, vo, so, AUX_SYS);
        __builtin_amdgcn_raw_buffer_store_b128(h[2 * t + 1], rsO, vo, so + 8 * NSH * 4, AUX_SYS);
      }
    }
    if (xi == NXP - 1) {
      const int vo = lane < NHQ ? lane * 16 : OOB;
      for (;;) {
        const v4u v = __builtin_amdgcn_raw_buffer_load_b128(rsS, vo, YF_OFF + w * YF_SRC, AUX_SYS);
        if (__all(lane >= NHQ || (v.x >= seq && v.y >= seq && v.z >= seq && v.w >= seq))) break;
        if (wt.give_up()) break;
        __builtin_amdgcn_s_sleep(2);
      }
    }
    if (wt.dead) __builtin_amdgcn_raw_buffer_store_b32(0x7fc00000u, rsO, vone, rank * (MSH * NSH * 4), AUX_SYS);
    return;
  }

  const bool helper = (qd & 4u) != 0u;
  const int hpan = (int)(qd & 3u), grp = (int)((qd >> 8) & 255u), ngrp = (int)((qd >> 16) & 255u), hq = (int)(qd >> 24);
  unsigned lst;
  int nun;
  {
    const int li = grp + lane * ngrp;
    const bool inr = li < (helper ? NHLP_U : NOWN_U);
    const unsigned e = a.list[helper ? 1 : 0][inr ? li : 0];
    const bool keep = inr && !(helper && (int)(e >> 5) == obase + hsrc);
    const unsigned long long km = __ballot(keep);
    nun = __builtin_popcountll(km);
    const int pos = keep ? (int)__builtin_amdgcn_mbcnt_hi((unsigned)(km >> 32), __builtin_amdgcn_mbcnt_lo((unsigned)km, 0u)) : 63;
    lst = (unsigned)__builtin_amdgcn_ds_permute(pos * 4, (int)e);
  }
  const unsigned dest = (unsigned)(helper ? obase + hsrc : rank);
  const int col0 = helper ? myidx * 128 + hpan * 32 : (hpan * 4 + hsrc) * 32;
  const char* const p_ws = a.ws[rank];
  const float* const p_out = a.out[dest & 7u];

  v4u b[64];
  v4u av[32];
  int eb[4];
  TAKE_SEQ();
  PUT_SEQ();
  __builtin_amdgcn_s_sleep(DELAY);
  {
    const int id = (hq * 4 + hsrc) * 4 + w;
    if (id < 256) {
      v4u h[16];
      const rsrc_t rs = mk_rsrc(helper ? a.x : a.w, X_BYTES);
      load_block(av, rs, helper ? x_vo(lane) : w_vo(lane), helper ? +X_VD : +W_VD, helper ? x_so(id >> 3, id & 7) : w_so(id & 15, (id >> 4) * 32),
                 helper ? +X_S0 : +W_S0, helper ? +X_S1 : +W_S1);
      int se;
      if (helper) se = cvt_x(av, h, wlds, lane);
      else se = cvt_w(av, h);
      const int doff = helper ? XA_OFF + par * XA_PAR + rank * 32 * UNIT_B + id * ITEM_B : WB_OFF + par * WB_PAR + (256 + id) * ITEM_B;
      const int foff = helper ? XF_OFF + par * XF_PAR + (rank * 32 + (id >> 3)) * 128 + (id & 7) * 4
                              : WF_OFF + par * WF_PAR + (16 + (id >> 4)) * 128 + (id & 15) * 4;
#pragma unroll
      for (int s = 0; s < 16; ++s)
        __builtin_amdgcn_raw_buffer_store_b128(h[s], rsS, lane * 16 + (s & 3) * 1024, doff + (s >> 2) * 4096, AUX_SYS);
      WAIT_VM(0);
      __builtin_amdgcn_raw_buffer_store_b32(seq * 256u + (unsigned)se, rsS, vone, foff, AUX_SYS);
    }
    const int pidx = helper ? hsrc * 4 + hpan : 16 + hpan * 4 + hsrc;
    const int fo = WF_OFF + par * WF_PAR + pidx * 128 + w * 16;
    unsigned f0, f1, f2, f3;
    for (;;) {
      const v4u v = __builtin_amdgcn_raw_buffer_load_b128(rsS, 0, fo, AUX_SYS);
      f0 = rfl(v.x); f1 = rfl(v.y); f2 = rfl(v.z); f3 = rfl(v.w);
      if ((f0 >> 8) == seq && (f1 >> 8) == seq && (f2 >> 8) == seq && (f3 >> 8) == seq) break;
      if (wt.give_up()) break;
      __builtin_amdgcn_s_sleep(8);
    }
    eb[0] = 254 - (int)(f0 & 255u); eb[1] = 254 - (int)(f1 & 255u); eb[2] = 254 - (int)(f2 & 255u); eb[3] = 254 - (int)(f3 & 255u);
    const int bo = WB_OFF + par * WB_PAR + (pidx * 16 + w * 4) * ITEM_B;
#pragma unroll
    for (int s = 0; s < 64; ++s)
      b[s] = __builtin_amdgcn_raw_buffer_load_b128(rsS, lane * 16 + (s & 3) * 1024, bo + (s >> 2) * 4096, AUX_SYS);
    SCHED_FENCE();
  }
  __syncthreads();

#pragma unroll
  for (int s = 0; s < NAGPR; ++s) asm volatile("" : "+a"(b[s]));

  const int l_a = XA_OFF + par * XA_PAR + w * (2 * ITEM_B) + (int)lst * UNIT_B;
  const int l_y = (int)lst * (16 * NSH * 4) + col0 * 4;
  const int l_f = XF_OFF + par * XF_PAR + w * 8 + (int)lst * 128;
  const unsigned l_d = seq << 12 | lst << 2 | (unsigned)hpan;
  const unsigned stmask = (unsigned)__ballot(
      helper && !((int)(lst >> 7) == (rank >> 2) && (int)(lst >> 5) != rank && (int)(lst & 31u) >= KU - NDIR));
  int vk[8];
#pragma unroll
  for (int k = 0; k < 8; ++k) vk[k] = lane * 16 + k * 4096;
  auto pin_vk = [&]() INL {
    asm volatile("" : "+v"(vk[0]), "+v"(vk[1]), "+v"(vk[2]), "+v"(vk[3]), "+v"(vk[4]), "+v"(vk[5]), "+v"(vk[6]), "+v"(vk[7]));
  };
  auto load_a = [&](int s, int ao) INL { return __builtin_amdgcn_raw_buffer_load_b128(rsS, vk[s >> 2] + (s & 3) * 1024, ao, AUX_SYS); };

  unsigned todo = (1u << nun) - 1u, watch = todo;
  unsigned rdy = 0, part = 0, mine = 0;
  unsigned exps = 0;
  float bad = wt.dead ? __builtin_nanf("") : 0.f;
  if (wt.dead) mine = 0x7fffffffu;
  const unsigned seq8 = seq << 8;
  v2u pv;
  auto poll_issue = [&]() INL {
    const int lf = l_f;
    pv = __builtin_amdgcn_raw_buffer_load_b64(rsS, __builtin_amdgcn_inverse_ballot_w64(watch & ~mine) ? lf : +OOB, 0, AUX_SYS);
  };
  auto poll_eval = [&]() INL {
    const bool ok = ((pv.x ^ seq8) | (pv.y ^ seq8)) < 256u;
    exps = ok ? __builtin_amdgcn_perm(pv.y, pv.x, 0x0c0c0400u) : exps;
    mine |= (unsigned)__ballot(ok);
  };

  int ld_w = w * RED_WAVE + (lane & 15) * 144 + ((lane >> 4) & 1) * 64 + (lane >> 5) * 8;
  int ld_r = (w * 4 + ((lane >> 3) & 3)) * 144 + (lane & 7) * 16;
  int ld_c = RED_BUF;
  const int vy = lane < 32 ? (w * 4 + (lane >> 3)) * (NSH * 4) + (lane & 7) * 16 : OOB;
  const int vs = lane < 32 ? w * 512 + lane * 16 : OOB;
  const int ylink = (hsrc - myidx) & 3;
  const unsigned ybase = (seq - 1u) * (unsigned)NST;
  const int ys0 = YS_OFF + ylink * YS_LINK, yd0 = YD_OFF + ylink * YD_LINK + w * 4;
  int vzero = 0;
  asm volatile("" : "+v"(vzero));
  int pend_vo = OOB;
  unsigned pend_val = 0;
  auto emit = [&]() INL {
    __builtin_amdgcn_raw_buffer_store_b32(pend_val, rsS, pend_vo, yd0, AUX_SYS);
    pend_vo = OOB;
  };
  auto post = [&](unsigned slot) INL { *(LAS v2u*)((LAS char*)lds + ld_c + w * 8) = v2u{mine, slot}; };
  auto share = [&](v4u m0, v4u m1) INL {
    rdy = rfl(m0.x & m0.z & m1.x & m1.z);
    part = rfl(m0.x | m0.z | m1.x | m1.z);
    ld_w ^= 16384;
    ld_r ^= 16384;
    ld_c ^= 16384;
  };
  auto choose = [&]() INL {
    const unsigned c2 = part & todo;
    unsigned c = rdy & todo;
    c = c ? c : c2;
    c = __builtin_popcount(todo) == 1 ? todo : c;
    const int i = __builtin_ctz(c | 0x80000000u);
    todo &= ~(1u << i);
    return i;
  };
  auto have = [&](int i) INL { return (mine >> i & 1u) != 0u; };
  auto repoll = [&]() INL {
    if (wt.give_up()) {
      mine = 0x7fffffffu;
      bad = __builtin_nanf("");
    }
    __builtin_amdgcn_s_sleep(1);
    poll_issue();
    WAIT_VM(1);
    emit();
  };

  int o_st = 0, o_so = 0;
  unsigned o_d = 0;
  auto front = [&](auto pf_tag, int icur, int inext) INL {
    constexpr bool PF = decltype(pf_tag)::value;
    pin_vk();
    const int prev_st = o_st;
    int ao = 0, tslot = 0, ex[4];
    f32x4 acc0 = {0.f, 0.f, 0.f, 0.f}, acc1 = {0.f, 0.f, 0.f, 0.f};
    f32x2 p[4];
#pragma unroll
    for (int j = 0; j < 4; ++j) p[j] = f32x2{bad, bad};
#pragma unroll
    for (int s = 0; s < 32; ++s) {
      if (s > 0 && (s & 7) == 0 && ex[(s >> 3) - 1] != ex[s >> 3]) {
#pragma unroll
        for (int j = 0; j < 4; ++j) {
          p[j][0] += __builtin_amdgcn_ldexpf(acc0[j], ex[(s >> 3) - 1]);
          p[j][1] += __builtin_amdgcn_ldexpf(acc1[j], ex[(s >> 3) - 1]);
        }
        acc0 = acc1 = f32x4{0.f, 0.f, 0.f, 0.f};
      }
      const v4u cur = av[s];
      acc0 = mfma(b[2 * s], cur, acc0);
      acc1 = mfma(b[2 * s + 1], cur, acc1);
      if (s == 0 && PF) {
        watch &= ~(1u << inext);
        ao = __builtin_amdgcn_readlane(l_a, inext);
        poll_issue();
      }
      if (s == 1) {
        o_st = (int)(stmask >> icur & 1u);
        if (o_st && threadIdx.x == 0) tslot = __builtin_amdgcn_raw_ptr_buffer_atomic_add_i32(1, rsS, vzero, YQ_OFF + ylink * 128, 0);
      }
      if (s == 3) {
        const unsigned ex2 = (unsigned)__builtin_amdgcn_readlane((int)exps, icur);
        const int ea0 = (int)(ex2 & 255u), ea1 = (int)(ex2 >> 8);
        ex[0] = eb[0] - ea0; ex[1] = eb[1] - ea0; ex[2] = eb[2] - ea1; ex[3] = eb[3] - ea1;
        asm volatile("" : "+s"(ex[0]), "+s"(ex[1]), "+s"(ex[2]), "+s"(ex[3]));
      }
      if (s == 5) {
        o_so = __builtin_amdgcn_readlane(l_y, icur);
        o_d = (unsigned)__builtin_amdgcn_readlane((int)l_d, icur);
        asm volatile("" : "+s"(o_so), "+s"(o_d));
      }
      if (PF && s >= 2 && (s - 1) * NLOOP / 30 > (s - 2) * NLOOP / 30) av[(s - 2) * NLOOP / 30] = load_a((s - 2) * NLOOP / 30, ao);
      if (!PF && s == POLL_AT) poll_issue();
      if (PF && s == 29) {
        WAIT_VM(28 * NLOOP / 30);
        poll_eval();
        emit();
        post(rfl((unsigned)tslot) - ybase);
      }
      SCHED_FENCE();
    }
#pragma unroll
    for (int j = 0; j < 4; ++j) {
      if (PF && j < NPRE) av[NLOOP + j] = load_a(NLOOP + j, ao);
      SCHED_FENCE();
      p[j][0] += __builtin_amdgcn_ldexpf(acc0[j], ex[3]);
      p[j][1] += __builtin_amdgcn_ldexpf(acc1[j], ex[3]);
      *(LAS f32x2*)((LAS char*)lds + ld_w + j * 16) = p[j];
      SCHED_FENCE();
    }
    if (!PF && (watch != 0u || o_st || prev_st)) {
      WAIT_VM(0);
      poll_eval();
      emit();
      post(rfl((unsigned)tslot) - ybase);
    }
    SCHED_FENCE();
    return ao;
  };
  auto back = [&](auto pf_tag, int ao) INL {
    constexpr bool PF = decltype(pf_tag)::value;
    constexpr int NB = 32 - NLOOP - NPRE;
    auto more = [&](auto k0, auto k1) INL {
      SCHED_FENCE();
      if (PF) {
#pragma unroll
        for (int s = NLOOP + NPRE + NB * decltype(k0)::value / 4; s < NLOOP + NPRE + NB * decltype(k1)::value / 4; ++s) av[s] = load_a(s, ao);
      }
      SCHED_FENCE();
    };
#define MORE(k0, k1) more(std::integral_constant<int, k0>(), std::integral_constant<int, k1>())
    LDS_BARRIER();
    SCHED_FENCE();
    const LAS char* const rb = (const LAS char*)lds + ld_r;
    const v4u m0 = *(const LAS v4u*)((LAS char*)lds + ld_c), m1 = *(const LAS v4u*)((LAS char*)lds + ld_c + 16);
    const f32x4 p0 = *(const LAS f32x4*)rb, p1 = *(const LAS f32x4*)(rb + RED_WAVE);
    const f32x4 p2 = *(const LAS f32x4*)(rb + 2 * RED_WAVE), p3 = *(const LAS f32x4*)(rb + 3 * RED_WAVE);
    MORE(0, 1);
    const int st = (int)rfl((unsigned)o_st), vs0 = vs, vy0 = vy, vone0 = vone;
    const rsrc_t rs = mk_rsrc(st ? (const void*)p_ws : (const void*)p_out, st ? WS_USED : OUT_BYTES);
    int vo = st ? vs0 : vy0;
    pend_vo = st ? vone0 : +OOB;
    pend_val = o_d;
    asm volatile("" : "+v"(vo), "+v"(pend_vo), "+v"(pend_val));
    MORE(1, 2);
    share(m0, m1);
    const int ts = (int)(rfl(m0.y) & 511u);
    int so = (int)rfl((unsigned)(st ? ys0 + ts * 2048 : +o_so));
    pend_vo = (int)((unsigned)pend_vo + (unsigned)ts * 16u);
    asm volatile("" : "+s"(rdy), "+s"(part), "+s"(so));
    MORE(2, 3);
    f32x4 y = (p0 + p1) + (p2 + p3);
    y.x = y.x <= 0.f ? 0.f : y.x;
    y.y = y.y <= 0.f ? 0.f : y.y;
    y.z = y.z <= 0.f ? 0.f : y.z;
    y.w = y.w <= 0.f ? 0.f : y.w;
    __builtin_amdgcn_raw_buffer_store_b128(__builtin_bit_cast(v4u, y), rs, vo, so, AUX_SYS);
    MORE(3, 4);
  };

  int icur, ci = 31;
  bool defer = false;
  poll_issue();
  emit();
  for (;;) {
    while (ci == 31) {
      poll_eval();
      post(0u);
      LDS_BARRIER();
      share(*(const LAS v4u*)((LAS char*)lds + ld_c), *(const LAS v4u*)((LAS char*)lds + ld_c + 16));
      ci = choose();
      if (!have(ci)) repoll();
    }
    while (!have(ci)) {
      poll_eval();
      if (!have(ci)) repoll();
    }
    icur = ci;
    watch &= ~(1u << icur);
    {
      const int ao = __builtin_amdgcn_readlane(l_a, icur);
      pin_vk();
#pragma unroll
      for (int s = 0; s < 32; ++s) {
        av[s] = load_a(s, ao);
        SCHED_FENCE();
      }
    }
    if (defer) {
      back(std::false_type(), 0);
    } else {
      WAIT_VM(32);
      emit();
    }
    ci = choose();
    while (have(ci)) {
      const int ao = front(std::true_type(), icur, ci);
      back(std::true_type(), ao);
      icur = ci;
      ci = choose();
    }
    front(std::false_type(), icur, 0);
    defer = have(ci);
    if (!defer) {
      if (watch != 0u) poll_issue();
      back(std::false_type(), 0);
      if (watch == 0u) break;
      if (ci == 31) ci = choose();
    }
  }
  WAIT_VM(0);
  emit();
  if (helper) {
    const rsrc_t rsD = mk_rsrc(a.ws[dest & 7u], WS_USED);
    __builtin_amdgcn_raw_buffer_store_b32(seq, rsD, vone, YF_OFF + myidx * YF_SRC + (hq * 4 + w) * 4, AUX_SYS);
  }
}

struct Ev { double t; int e; };

}

extern "C" void kernel_launch(int rank, int nranks, ncclComm_t comm,
                              hipStream_t stream,
                              void* const* d_in, const int* in_sizes, int n_in,
                              void* d_out, int out_size,
                              const int* in_dtypes, int out_dtype,
                              void* const* d_out_all,
                              void* const* d_ws_all, size_t ws_bytes) {
  (void)comm; (void)in_dtypes; (void)out_dtype; (void)d_out;
  if (nranks != NRANK || n_in < 2 || (unsigned)rank >= (unsigned)NRANK || ws_bytes < (size_t)WS_USED ||
      in_sizes[0] != MSH * KDIM || in_sizes[1] != KDIM * NSH || out_size != NRANK * MSH * NSH) {
    fprintf(stderr, "unexpected problem shape\n");
    return;
  }
  Args a = {};
  a.x = (const float*)d_in[0];
  a.w = (const float*)d_in[1];
  for (int p = 0; p < NRANK; ++p) {
    a.ws[p] = (char*)d_ws_all[p];
    a.out[p] = (float*)d_out_all[p];
  }
  a.rank = rank;

  const int gbase = rank & 4, obase = gbase ^ 4;
  std::vector<Ev> ev;
  for (int u = 0; u < 32; ++u) ev.push_back({0.001 * u, rank * 32 + u});
  for (int p = gbase; p < gbase + 4; ++p)
    if (p != rank)
      for (int u = 0; u < KU; ++u) ev.push_back({1.0 + u + 0.01 * p, p * 32 + u});
  for (int p = obase; p < obase + 4; ++p)
    for (int v = 0; v < GU; ++v) ev.push_back({9.1 + v + 0.01 * p, p * 32 + KU + v});
  std::stable_sort(ev.begin(), ev.end(), [](const Ev& l, const Ev& r) { return l.t < r.t; });
  int n0 = 0, n1 = 0;
  for (const Ev& v : ev) {
    a.list[0][n0++] = (unsigned char)v.e;
    if (!((v.e >> 5) == rank && (v.e & 31) >= KU)) a.list[1][n1++] = (unsigned char)v.e;
  }
  const int gh[4] = {6, 6, 6, 8};
  static_assert(NHQ <= 32 && NHQ >= 16 && GOWN >= 4, "done flags of one helper fit 512 bytes; 256 blocks of x and of w are converted");
  if (gh[0] + gh[1] + gh[2] + gh[3] != NHQ || n0 != NOWN_U || n1 != NHLP_U) { fprintf(stderr, "bad quad split\n"); return; }
  int used[8] = {0, 0, 0, 0, 0, 0, 0, 0};
  for (int c = 0; c < 4; ++c)
    for (int g = 0; g < GOWN; ++g) {
      const int x = c + 4 * (g & 1);
      a.quad[x * 6 + used[x]++] = (unsigned)c | (unsigned)g << 8 | (unsigned)GOWN << 16 | (unsigned)(g * 4 + c) << 24;
    }
  for (int q = 0; q < NXQ; ++q) a.quad[(7 - q) * 6 + used[7 - q]++] = 8u | (unsigned)q << 24;
  int hq = 0, x = 0;
  for (int p = 3; p >= 0; --p)
    for (int g = 0; g < gh[p]; ++g) {
      if ((NHLP_U + gh[p] - 1) / gh[p] > LMAX) { fprintf(stderr, "bad quad split\n"); return; }
      while (used[x] >= 6) x = (x + 1) & 7;
      a.quad[x * 6 + used[x]++] = (unsigned)(4 + p) | (unsigned)g << 8 | (unsigned)gh[p] << 16 | (unsigned)hq << 24;
      ++hq;
      x = (x + 1) & 7;
    }
  static_assert((NOWN_U + GOWN - 1) / GOWN <= LMAX, "one list entry per bit");
  hipLaunchKernelGGL(ag_gemm_kernel, dim3(256), dim3(256), 0, stream, a);
  HIP_CHECK(hipGetLastError());
}
```
